```python
import numpy as np

kernel_name = "gpu_gemm_rs_m4096_k4096_n16384_f32_reluq8"


ABSMAX_RTOL = 0.02

COMPUTE_DTYPE = "float32"
EPILOGUE = "relu_quant_int8"
IN_QUANT = None
OUT_QUANT = "int8"
IN_DTYPES = {"x": "float32", "w_mat": "float32"}
WORLD_SIZE = 8

def setup_inputs():

    x = np.random.randn(4096, 4096).astype(np.float32)
    w_mat = (np.random.randn(4096, 16384) / np.sqrt(4096)).astype(np.float32)
    return {"x": x, "w_mat": w_mat}

def _quant_dequant(y, amax):
    scale = amax / 127.0
    q = np.clip(np.round(y / scale), -127.0, 127.0)
    return (q * scale).astype(np.float32)
def _epilogue(y):
    y = np.maximum(y, 0.0)
    return _quant_dequant(y, np.abs(y).max())

def reference(x, w_mat):
    y = (x @ w_mat).astype(np.float32)
    return _epilogue(y)
```

```cpp
#include <hip/hip_runtime.h>
#include <rccl/rccl.h>
#include <cstdio>

#define HIP_CHECK(expr) do { hipError_t _e = (expr); if (_e != hipSuccess) { \
    fprintf(stderr, "HIP error %s at %s:%d\n", hipGetErrorString(_e), __FILE__, __LINE__); \
    return; } } while (0)

namespace {

typedef short bf16x8 __attribute__((ext_vector_type(8)));
typedef __bf16 bf16x2_t __attribute__((ext_vector_type(2)));
typedef float f32x4 __attribute__((ext_vector_type(4)));
typedef float f32x2 __attribute__((ext_vector_type(2)));
typedef unsigned u32x4 __attribute__((ext_vector_type(4)));
typedef unsigned u32x2 __attribute__((ext_vector_type(2)));
#define LAS __attribute__((address_space(3)))
#define GAS __attribute__((address_space(1)))

constexpr int NR = 8;
constexpr int MG = 4096, KSH = 512, NG = 16384, NBLK = 2048, MSH = 512;
constexpr unsigned PIECE = 32768;
constexpr unsigned STEP_BYTES = 24 * PIECE;
constexpr size_t DATA_BYTES = (size_t)NR * 8 * STEP_BYTES;
constexpr size_t SRC_BYTES = (size_t)8 * STEP_BYTES;
constexpr size_t FLAG_OFF = DATA_BYTES;
constexpr size_t FLAG_BYTES = (size_t)NR * 8 * 48 * 128;
constexpr size_t GRAN_OFF = FLAG_OFF + FLAG_BYTES;
constexpr size_t DONE_OFF = GRAN_OFF + 8192;
constexpr size_t CNT_OFF = DONE_OFF + 4096;
constexpr size_t TKT_OFF = CNT_OFF + 256 * 128;
constexpr size_t TMO_OFF = TKT_OFF + 4096;
constexpr size_t DBG_OFF = TMO_OFF + 4096;
constexpr size_t WS_NEED = DBG_OFF + 256 * 1024;
constexpr int NPUSH = 64, NGEMM = 128, NWG = NPUSH + NGEMM;
constexpr unsigned long long DEADLINE = 50000000ull;
constexpr int SYS = 17;
constexpr int LDS_BYTES = 2 * 2 * PIECE + 1024;
constexpr unsigned NJOB = 384;
#define UNITS 2
#define TSTOP 300

struct Params {
  const float* x; const float* w;
  char* ws[NR]; char* out[NR];
  int rank;
};

__device__ __forceinline__ __amdgpu_buffer_rsrc_t mkrsrc(const void* p, unsigned bytes) {
  unsigned long long a = (unsigned long long)p;
  unsigned lo = __builtin_amdgcn_readfirstlane((unsigned)a), hi = __builtin_amdgcn_readfirstlane((unsigned)(a >> 32));
  void* q = (void*)(((unsigned long long)hi << 32) | lo);
  return __builtin_amdgcn_make_buffer_rsrc(q, 0, bytes, 0x00020000);
}
__device__ __forceinline__ char* pick(char* const (&t)[NR], int p) {
  char* r = t[0];
#pragma unroll
  for (int k = 1; k < NR; ++k) r = (k == p) ? t[k] : r;
  return r;
}
__device__ __forceinline__ unsigned pk(float a, float b) {
  bf16x2_t v = __builtin_convertvector((f32x2){a, b}, bf16x2_t);
  return __builtin_bit_cast(unsigned, v);
}
__device__ __forceinline__ unsigned ld_sys(const char* p) {
  return __hip_atomic_load((const GAS unsigned*)p, __ATOMIC_RELAXED, __HIP_MEMORY_SCOPE_SYSTEM);
}
__device__ __forceinline__ void st_sys(char* p, unsigned v) {
  __hip_atomic_store((GAS unsigned*)p, v, __ATOMIC_RELAXED, __HIP_MEMORY_SCOPE_SYSTEM);
}
__device__ __forceinline__ unsigned ld_agent(const char* p) {
  return __hip_atomic_load((const GAS unsigned*)p, __ATOMIC_RELAXED, __HIP_MEMORY_SCOPE_AGENT);
}
__device__ __forceinline__ void st_agent(char* p, unsigned v) {
  __hip_atomic_store((GAS unsigned*)p, v, __ATOMIC_RELAXED, __HIP_MEMORY_SCOPE_AGENT);
}
#define WAIT_V0() asm volatile("s_waitcnt vmcnt(0)" ::: "memory")
#define WAIT_L0() asm volatile("s_waitcnt lgkmcnt(0)" ::: "memory")
#define BAR() do { asm volatile("" ::: "memory"); __builtin_amdgcn_s_barrier(); asm volatile("" ::: "memory"); } while (0)
__device__ __forceinline__ bool give_up(char* tmo, unsigned call, unsigned long long t0) {
  bool dead = __builtin_amdgcn_readfirstlane(ld_agent(tmo)) == call || __builtin_amdgcn_s_memrealtime() - t0 > DEADLINE;
  if (dead) st_agent(tmo, call);
  return dead;
}
__device__ __forceinline__ unsigned wave_umax(unsigned v) {
  v = max(v, (unsigned)__builtin_amdgcn_update_dpp(0, (int)v, 0x111, 0xf, 0xf, true));
  v = max(v, (unsigned)__builtin_amdgcn_update_dpp(0, (int)v, 0x112, 0xf, 0xf, true));
  v = max(v, (unsigned)__builtin_amdgcn_update_dpp(0, (int)v, 0x114, 0xf, 0xf, true));
  v = max(v, (unsigned)__builtin_amdgcn_update_dpp(0, (int)v, 0x118, 0xf, 0xf, true));
  const unsigned a = __builtin_amdgcn_readlane(v, 15), b = __builtin_amdgcn_readlane(v, 31), c = __builtin_amdgcn_readlane(v, 47), d = __builtin_amdgcn_readlane(v, 63);
  return max(max(a, b), max(c, d));
}
#define STAMP(idx) do {} while (0)

__global__ void __launch_bounds__(512, 2) fused_gemm_q8(Params P) {
  __shared__ __attribute__((aligned(1024))) char smem[LDS_BYTES];
  LAS char* lds = (LAS char*)smem;
  LAS unsigned* misc = (LAS unsigned*)(lds + 4 * PIECE);
  const int tid = threadIdx.x, lane = tid & 63, wv = __builtin_amdgcn_readfirstlane(tid >> 6);
  const int b = blockIdx.x, xg = b & 7, slot = b >> 3;
  const int rank = P.rank;
  char* myws = pick(P.ws, rank);
  char* tmo = myws + TMO_OFF;
  STAMP(0);
  char* cntp = myws + CNT_OFF + (size_t)b * 128;
  const unsigned cv = ld_agent(cntp);

  if (slot < 8) {
    const int rel = slot, dst = (rank + rel) & 7;
    char* dws = pick(P.ws, dst);
    const __amdgpu_buffer_rsrc_t rx = mkrsrc(P.x, MG * KSH * 4), rw = mkrsrc(P.w, (unsigned)KSH * NG * 4u);
    const __amdgpu_buffer_rsrc_t rd = mkrsrc(dws + (size_t)rank * SRC_BYTES, (unsigned)SRC_BYTES);
    char* dflag = dws + FLAG_OFF + (size_t)rank * 8 * 48 * 128;
    const int g = lane >> 4, c16 = lane & 15;
    const unsigned xrot = 8u * ((8 - rel) & 3);
    const int v = wv & 3, unit = wv >> 2;
    const unsigned xlane = (unsigned)(v * 32 + c16) * 2048u + g * 16u;
    const unsigned wlane = (unsigned)((v & 1) * 32 + 4 * g) * 65536u + (unsigned)(dst * NBLK + (v >> 1) * 64 + 4 * c16) * 4u;
    f32x4 L[8];
    auto issue = [&](unsigned q) {
      const unsigned c = q / 48u, e = q % 48u;
      if (e < 16u) {
        const unsigned so = c * 64u * 65536u + e * 512u;
#pragma unroll
        for (int k = 0; k < 8; ++k)
          L[k] = __builtin_bit_cast(f32x4, __builtin_amdgcn_raw_buffer_load_b128(rw, wlane, so + (unsigned)(16 * (k >> 2) + (k & 3)) * 65536u, 0));
      } else {
        const unsigned hx = (e - 16u + xrot) & 31u;
        const unsigned so = hx * 128u * 2048u + c * 256u;
#pragma unroll
        for (int k = 0; k < 8; ++k)
          L[k] = __builtin_bit_cast(f32x4, __builtin_amdgcn_raw_buffer_load_b128(rx, xlane + (k & 3) * 64u, so + (unsigned)(k >> 2) * 32768u, 0));
      }
    };
    unsigned q = unit * 8 + xg, qn = (UNITS + unit) * 8 + xg;
    if (unit < UNITS) issue(q);
    LAS unsigned* tick = (LAS unsigned*)lds + unit * 1024;
    LAS unsigned* cnt = tick + 512;
    *(LAS u32x4*)(lds + tid * 16) = (u32x4){0u, 0u, 0u, 0u};
    const unsigned call = __builtin_amdgcn_readfirstlane(cv) + 1;
    asm volatile("s_waitcnt lgkmcnt(0)" :: "s"(call) : "memory");
    BAR();
    if (tid == 0) { st_agent(cntp, call); if (xg == 0) st_agent(myws + TKT_OFF + (size_t)(((call + 2) & 3) * 8 + rel) * 128, 0u); }
    GAS unsigned* tkt = (GAS unsigned*)(myws + TKT_OFF + (size_t)((call & 3) * 8 + rel) * 128);
    if (unit < UNITS) {
      unsigned prev_hp = 0, drawn = NJOB;
      const unsigned stop = unit == 0 ? NJOB : (unsigned)TSTOP;
      for (int r = 0;; ++r) {
        WAIT_V0();
        unsigned old = 0;
        if (r > 0) {
          if (lane == 0) old = __hip_atomic_fetch_add(cnt + r, 1u, __ATOMIC_RELAXED, __HIP_MEMORY_SCOPE_WORKGROUP);
          if (v == 0) {
            asm volatile("" : "+v"(drawn));
            qn = __builtin_amdgcn_readfirstlane(drawn) + 2u * UNITS * 8u;
            if (lane == 0) tick[r + 1] = qn + 1u;
          }
        }
        if (q >= NJOB) {
          if (__builtin_amdgcn_readfirstlane(old) == 3u && lane == 0) st_sys(dflag + (size_t)prev_hp * 128, call);
          break;
        }
        u32x4 o[4]; unsigned doff, hp;
        const unsigned c = q / 48u, e = q % 48u;
        if (e < 16u) {
#pragma unroll
          for (int t = 0; t < 4; ++t) o[t] = (u32x4){pk(L[0][t], L[1][t]), pk(L[2][t], L[3][t]), pk(L[4][t], L[5][t]), pk(L[6][t], L[7][t])};
          doff = 16u * PIECE + e * 16384u; hp = 32u + e;
        } else {
#pragma unroll
          for (int t = 0; t < 4; ++t) o[t] = (u32x4){pk(L[2 * t][0], L[2 * t][1]), pk(L[2 * t][2], L[2 * t][3]), pk(L[2 * t + 1][0], L[2 * t + 1][1]), pk(L[2 * t + 1][2], L[2 * t + 1][3])};
          const unsigned hx = (e - 16u + xrot) & 31u;
          doff = hx * 16384u; hp = hx;
        }
        asm volatile("" : "+v"(o[0]), "+v"(o[1]), "+v"(o[2]), "+v"(o[3]));
        if (__builtin_amdgcn_readfirstlane(old) == 3u && lane == 0) st_sys(dflag + (size_t)prev_hp * 128, call);
        doff += c * STEP_BYTES + (unsigned)v * 4096u + lane * 16u;
#pragma unroll
        for (int t = 0; t < 4; ++t) __builtin_amdgcn_raw_buffer_store_b128(o[t], rd, doff + t * 1024u, 0, SYS);
        prev_hp = c * 48u + hp;
        if (v != 0 && r > 0) {
          unsigned tv;
          while ((tv = __builtin_amdgcn_readfirstlane(__hip_atomic_load(tick + r + 1, __ATOMIC_RELAXED, __HIP_MEMORY_SCOPE_WORKGROUP))) == 0u) __builtin_amdgcn_s_sleep(1);
          qn = tv - 1u;
        }
        qn = __builtin_amdgcn_readfirstlane(qn);
        if (qn < NJOB) issue(qn);
        if (v == 0) {
          drawn = NJOB;
          if (qn < stop && lane == 0) {
            const unsigned one = 1u;
            asm volatile("global_atomic_add %0, %1, %2, off sc0" : "=v"(drawn) : "v"(tkt), "v"(one) : "memory");
          }
        }
        q = qn;
      }
    }
    STAMP(1);
    if (b == 0 && wv == 0) {
      const unsigned long long t0 = __builtin_amdgcn_s_memrealtime();
      const char* dn0 = myws + DONE_OFF + lane * 4;
      for (unsigned it = 1;; ++it) {
        const unsigned a0 = ld_sys(dn0), a1 = ld_sys(dn0 + 256);
        const bool ok = (a0 >= call || (lane >> 4) == rank) && (a1 >= call || (lane >> 4) + 4 == rank);
        if (__all(ok)) break;
        if ((it & 63u) == 0u && give_up(tmo, call, t0)) { if (lane == 0) pick(P.out, rank)[rank * NBLK] = (char)-128; break; }
        __builtin_amdgcn_s_sleep(1);
      }
      STAMP(2);
    }
    return;
  }

  const int sl = slot - 8, own = sl >> 1, ti = 2 * own + (xg & 1), tj = 2 * (xg >> 1) + (sl & 1);
  const int wr = wv >> 2, wc = wv & 3;
  unsigned order = 0;
  { int pos = 1;
#pragma unroll
    for (int q = 0; q < 8; ++q) { const int rl = (((ti >> 2) - (q >> 1)) & 3) + 4 * (q & 1); if (rl != 0) { order |= (unsigned)((rank + rl) & 7) << (4 * pos); ++pos; } }
    order |= (unsigned)rank; }
  const unsigned call = __builtin_amdgcn_readfirstlane(cv) + 1;
  const char* gx = myws + (size_t)ti * PIECE + wv * 4096 + lane * 16;
  const char* gw = myws + (size_t)(16 + tj) * PIECE + wv * 4096 + lane * 16;
  const char* fl = myws + FLAG_OFF + (size_t)((lane & 2) ? 32 + 2 * tj + (lane & 1) : 2 * ti + (lane & 1)) * 128;
  auto sidx = [&](int s) { return (int)((order >> (4 * (s & 7))) & 7u) * 8 + (s >> 3); };
  auto stage = [&](int s, int buf) {
    const size_t off = (size_t)sidx(s) * STEP_BYTES;
#define GLDS(src, dstoff, imm) __builtin_amdgcn_global_load_lds((const GAS unsigned*)(src), (LAS unsigned*)(lds + (dstoff)), 16, imm, SYS)
    GLDS(gx + off, buf * 2 * PIECE + wv * 4096, 0); GLDS(gx + off, buf * 2 * PIECE + wv * 4096, 1024);
    GLDS(gx + off, buf * 2 * PIECE + wv * 4096, 2048); GLDS(gx + off, buf * 2 * PIECE + wv * 4096, 3072);
    GLDS(gw + off, buf * 2 * PIECE + PIECE + wv * 4096, 0); GLDS(gw + off, buf * 2 * PIECE + PIECE + wv * 4096, 1024);
    GLDS(gw + off, buf * 2 * PIECE + PIECE + wv * 4096, 2048); GLDS(gw + off, buf * 2 * PIECE + PIECE + wv * 4096, 3072);
  };
  bool dead = false;
  auto wait_flag = [&](int s, unsigned fv) {
    if (__all(fv >= call)) return;
    const char* f = fl + (size_t)sidx(s) * 48 * 128;
    const unsigned long long t0 = __builtin_amdgcn_s_memrealtime();
    for (unsigned it = 1;; ++it) {
      if (__all(ld_sys(f) >= call)) return;
      if ((it & 63u) == 0u && give_up(tmo, call, t0)) return;
      __builtin_amdgcn_s_sleep(1);
    }
  };
  f32x4 acc[8][4];
#pragma unroll
  for (int m = 0; m < 8; ++m)
#pragma unroll
    for (int t = 0; t < 4; ++t) acc[m][t] = (f32x4){0.f, 0.f, 0.f, 0.f};
  unsigned xo[2], wo[2];
  xo[0] = wr * 16384 + lane * 16; wo[0] = PIECE + wc * 8192 + lane * 16; xo[1] = xo[0] + 2 * PIECE; wo[1] = wo[0] + 2 * PIECE;
  asm volatile("" : "+v"(xo[0]), "+v"(wo[0]), "+v"(xo[1]), "+v"(wo[1]));
  if (tid < 32) misc[tid] = 0;
  unsigned fv = 0;
  if (wv == 0) wait_flag(0, 0u);
  asm volatile("s_waitcnt lgkmcnt(0)" :: "s"(call) : "memory");
  BAR();
  if (tid == 0) st_agent(cntp, call);
  STAMP(1);
  stage(0, 0);
  if (wv == 0) fv = ld_sys(fl + (size_t)sidx(1) * 48 * 128);
#pragma unroll 2
  for (int s = 0; s < 64; ++s) {
    const int buf = s & 1;
    WAIT_V0();
    if (wv == 0 && s + 1 < 64) wait_flag(s + 1, fv);
    BAR();
    __builtin_amdgcn_s_waitcnt(0xC07F);
    if (s + 1 < 64) stage(s + 1, buf ^ 1);
    if (wv == 0 && s + 2 < 64) fv = ld_sys(fl + (size_t)sidx(s + 2) * 48 * 128);
    bf16x8 X[2][8], W[2][4];
#pragma unroll
    for (int h = 0; h < 2; ++h) {
#pragma unroll
      for (int t = 0; t < 4; ++t) W[h][t] = *(const LAS bf16x8*)(lds + wo[buf] + h * 4096 + t * 1024);
#pragma unroll
      for (int m = 0; m < 8; ++m) X[h][m] = *(const LAS bf16x8*)(lds + xo[buf] + m * 2048 + h * 1024);
    }
#pragma unroll
    for (int h = 0; h < 2; ++h)
#pragma unroll
      for (int m = 0; m < 8; ++m)
#pragma unroll
        for (int t = 0; t < 4; ++t) acc[m][t] = __builtin_amdgcn_mfma_f32_16x16x32_bf16(W[h][t], X[h][m], acc[m][t], 0, 0, 0);
    __builtin_amdgcn_sched_group_barrier(0x100, 8, 0);
#pragma unroll
    for (int k = 0; k < 16; ++k) { __builtin_amdgcn_sched_group_barrier(0x008, 2, 0); __builtin_amdgcn_sched_group_barrier(0x100, 1, 0); }
    __builtin_amdgcn_sched_group_barrier(0x008, 32, 0);
    __builtin_amdgcn_sched_barrier(0);
  }
  STAMP(2);
  float mx = 0.f;
#pragma unroll
  for (int m = 0; m < 8; ++m)
#pragma unroll
    for (int t = 0; t < 4; ++t) { mx = __builtin_fmaxf(__builtin_fmaxf(mx, acc[m][t][0]), acc[m][t][1]); mx = __builtin_fmaxf(__builtin_fmaxf(mx, acc[m][t][2]), acc[m][t][3]); }
  {
    const unsigned wm = wave_umax(__float_as_uint(mx));
    unsigned old = 0;
    if (lane == 0) {
      __hip_atomic_fetch_max(misc + 0, wm, __ATOMIC_RELAXED, __HIP_MEMORY_SCOPE_WORKGROUP);
      old = __hip_atomic_fetch_add(misc + 1, 1u, __ATOMIC_RELAXED, __HIP_MEMORY_SCOPE_WORKGROUP);
    }
    if (__builtin_amdgcn_readfirstlane(old) == 7u) {
      const unsigned mb = __hip_atomic_load(misc + 0, __ATOMIC_RELAXED, __HIP_MEMORY_SCOPE_WORKGROUP);
      if (lane < NR) {
        char* gp = P.ws[0];
#pragma unroll
        for (int k = 1; k < NR; ++k) gp = (lane == k) ? P.ws[k] : gp;
        __hip_atomic_store((GAS unsigned long long*)(gp + GRAN_OFF + (size_t)(rank * 128 + ti * 8 + tj) * 8), ((unsigned long long)call << 32) | mb,
                           __ATOMIC_RELAXED, __HIP_MEMORY_SCOPE_SYSTEM);
      }
    }
  }
  {
    const __amdgpu_buffer_rsrc_t rg = mkrsrc(myws + GRAN_OFF, 8192);
    const unsigned long long t0 = __builtin_amdgcn_s_memrealtime();
    unsigned mb = 0;
    for (unsigned it = 1;; ++it) {
      const u32x4 gq = __builtin_amdgcn_raw_buffer_load_b128(rg, wv * 1024 + lane * 16, 0, SYS);
      const unsigned tv = ld_agent(tmo);
      mb = max(gq.x, gq.z);
      dead = __builtin_amdgcn_readfirstlane(tv) == call;
      if (dead || __all(gq.y == call && gq.w == call)) break;
      if ((it & 63u) == 0u && give_up(tmo, call, t0)) { dead = true; break; }
      __builtin_amdgcn_s_sleep(1);
      asm volatile("" ::: "memory");
    }
    mb = wave_umax(mb);
    if (lane == 0) { __hip_atomic_fetch_max(misc + 2, mb, __ATOMIC_RELAXED, __HIP_MEMORY_SCOPE_WORKGROUP); if (dead) misc[3] = 1u; }
  }
  WAIT_L0();
  BAR();
  STAMP(3);
  const float amax = __uint_as_float(misc[2]);
  const bool bad = misc[3] != 0u;
  const float scale = amax / 127.0f;
  const float inv = scale > 0.f ? 1.0f / scale : 0.f;
  const __amdgpu_buffer_rsrc_t ro = mkrsrc(pick(P.out, own), (unsigned)MSH * NG);
  const unsigned obase = (unsigned)((ti & 1) * 256 + wr * 128 + (lane & 15)) * (unsigned)NG + (unsigned)(rank * NBLK + tj * 256 + wc * 64 + (lane >> 4) * 16);
#pragma unroll
  for (int m = 0; m < 8; ++m) {
    u32x4 q;
#pragma unroll
    for (int j = 0; j < 4; ++j) {
      unsigned by[4];
#pragma unroll
      for (int t = 0; t < 4; ++t) by[t] = __float_as_uint(__builtin_amdgcn_fmed3f(acc[m][t][j] * inv, 0.f, 127.0f) + 12582912.0f);
      const unsigned lo = __builtin_amdgcn_perm(by[1], by[0], 0x0c0c0400u), hi = __builtin_amdgcn_perm(by[3], by[2], 0x04000c0cu);
      q[j] = bad ? 0x80808080u : (lo | hi);
    }
    __builtin_amdgcn_raw_buffer_store_b128(q, ro, obase + (unsigned)m * 16u * NG, 0, SYS);
  }
  WAIT_V0();
  {
    unsigned old = 0;
    if (lane == 0) old = __hip_atomic_fetch_add(misc + 4, 1u, __ATOMIC_RELAXED, __HIP_MEMORY_SCOPE_WORKGROUP);
    if (__builtin_amdgcn_readfirstlane(old) == 7u && lane == 0 && own != rank)
      st_sys(pick(P.ws, own) + DONE_OFF + (size_t)(rank * 16 + (ti & 1) * 8 + tj) * 4, call);
  }
  STAMP(4);
}

}

extern "C" void kernel_launch(int rank, int nranks, ncclComm_t comm, hipStream_t stream,
                              void* const* d_in, const int* in_sizes, int n_in,
                              void* d_out, int out_size,
                              const int* in_dtypes, int out_dtype,
                              void* const* d_out_all,
                              void* const* d_ws_all, size_t ws_bytes) {
  if (nranks != NR || (unsigned)rank >= (unsigned)NR || n_in != 2 || in_sizes[0] != MG * KSH || in_sizes[1] != KSH * NG ||
      out_size != MSH * NG || ws_bytes < WS_NEED) {
    fprintf(stderr, "unexpected problem shape\n");
    return;
  }
  Params P;
  P.x = (const float*)d_in[0]; P.w = (const float*)d_in[1]; P.rank = rank;
  for (int p = 0; p < NR; ++p) { P.ws[p] = (char*)d_ws_all[p]; P.out[p] = (char*)d_out_all[p]; }
  hipLaunchKernelGGL(fused_gemm_q8, dim3(NWG), dim3(512), 0, stream, P);
  HIP_CHECK(hipGetLastError());
}
```

```cpp
#include <hip/hip_runtime.h>
#include <rccl/rccl.h>
#include <cstdio>

#define HIP_CHECK(expr) do { hipError_t _e = (expr); if (_e != hipSuccess) { \
    fprintf(stderr, "HIP error %s at %s:%d\n", hipGetErrorString(_e), __FILE__, __LINE__); \
    return; } } while (0)

namespace {

typedef short bf16x8 __attribute__((ext_vector_type(8)));
typedef __bf16 bf16x2_t __attribute__((ext_vector_type(2)));
typedef float f32x4 __attribute__((ext_vector_type(4)));
typedef float f32x2 __attribute__((ext_vector_type(2)));
typedef unsigned u32x4 __attribute__((ext_vector_type(4)));
typedef unsigned u32x2 __attribute__((ext_vector_type(2)));
#define LAS __attribute__((address_space(3)))
#define GAS __attribute__((address_space(1)))

constexpr int NR = 8;
constexpr int MG = 4096, KSH = 512, NG = 16384, NBLK = 2048, MSH = 512;
constexpr unsigned PIECE = 32768;
constexpr unsigned STEP_BYTES = 24 * PIECE;
constexpr size_t DATA_BYTES = (size_t)NR * 8 * STEP_BYTES;
constexpr size_t SRC_BYTES = (size_t)8 * STEP_BYTES;
constexpr size_t FLAG_OFF = DATA_BYTES;
constexpr size_t FLAG_BYTES = (size_t)NR * 8 * 48 * 128;
constexpr size_t GRAN_OFF = FLAG_OFF + FLAG_BYTES;
constexpr size_t DONE_OFF = GRAN_OFF + 8192;
constexpr size_t CNT_OFF = DONE_OFF + 4096;
constexpr size_t TKT_OFF = CNT_OFF + 256 * 128;
constexpr size_t TMO_OFF = TKT_OFF + 4096;
constexpr size_t DBG_OFF = TMO_OFF + 4096;
constexpr size_t WS_NEED = DBG_OFF + 256 * 1024;
constexpr int NPUSH = 128, NGEMM = 128, NWG = NPUSH + NGEMM;
constexpr unsigned long long DEADLINE = 50000000ull;
constexpr int SYS = 17;
constexpr int LDS_BYTES = 2 * 2 * PIECE + 1024;
constexpr unsigned NJOB = 384;
#define UNITS 1
#define TSTOP 300

struct Params {
  const float* x; const float* w;
  char* ws[NR]; char* out[NR];
  int rank;
};

__device__ __forceinline__ __amdgpu_buffer_rsrc_t mkrsrc(const void* p, unsigned bytes) {
  unsigned long long a = (unsigned long long)p;
  unsigned lo = __builtin_amdgcn_readfirstlane((unsigned)a), hi = __builtin_amdgcn_readfirstlane((unsigned)(a >> 32));
  void* q = (void*)(((unsigned long long)hi << 32) | lo);
  return __builtin_amdgcn_make_buffer_rsrc(q, 0, bytes, 0x00020000);
}
__device__ __forceinline__ char* pick(char* const (&t)[NR], int p) {
  char* r = t[0];
#pragma unroll
  for (int k = 1; k < NR; ++k) r = (k == p) ? t[k] : r;
  return r;
}
__device__ __forceinline__ unsigned pk(float a, float b) {
  bf16x2_t v = __builtin_convertvector((f32x2){a, b}, bf16x2_t);
  return __builtin_bit_cast(unsigned, v);
}
__device__ __forceinline__ unsigned ld_sys(const char* p) {
  return __hip_atomic_load((const GAS unsigned*)p, __ATOMIC_RELAXED, __HIP_MEMORY_SCOPE_SYSTEM);
}
__device__ __forceinline__ void st_sys(char* p, unsigned v) {
  __hip_atomic_store((GAS unsigned*)p, v, __ATOMIC_RELAXED, __HIP_MEMORY_SCOPE_SYSTEM);
}
__device__ __forceinline__ unsigned ld_agent(const char* p) {
  return __hip_atomic_load((const GAS unsigned*)p, __ATOMIC_RELAXED, __HIP_MEMORY_SCOPE_AGENT);
}
__device__ __forceinline__ void st_agent(char* p, unsigned v) {
  __hip_atomic_store((GAS unsigned*)p, v, __ATOMIC_RELAXED, __HIP_MEMORY_SCOPE_AGENT);
}
#define WAIT_V0() asm volatile("s_waitcnt vmcnt(0)" ::: "memory")
#define WAIT_L0() asm volatile("s_waitcnt lgkmcnt(0)" ::: "memory")
#define BAR() do { asm volatile("" ::: "memory"); __builtin_amdgcn_s_barrier(); asm volatile("" ::: "memory"); } while (0)
__device__ __forceinline__ bool give_up(char* tmo, unsigned call, unsigned long long t0) {
  bool dead = __builtin_amdgcn_readfirstlane(ld_agent(tmo)) == call || __builtin_amdgcn_s_memrealtime() - t0 > DEADLINE;
  if (dead) st_agent(tmo, call);
  return dead;
}
__device__ __forceinline__ unsigned wave_umax(unsigned v) {
  v = max(v, (unsigned)__builtin_amdgcn_update_dpp(0, (int)v, 0x111, 0xf, 0xf, true));
  v = max(v, (unsigned)__builtin_amdgcn_update_dpp(0, (int)v, 0x112, 0xf, 0xf, true));
  v = max(v, (unsigned)__builtin_amdgcn_update_dpp(0, (int)v, 0x114, 0xf, 0xf, true));
  v = max(v, (unsigned)__builtin_amdgcn_update_dpp(0, (int)v, 0x118, 0xf, 0xf, true));
  const unsigned a = __builtin_amdgcn_readlane(v, 15), b = __builtin_amdgcn_readlane(v, 31), c = __builtin_amdgcn_readlane(v, 47), d = __builtin_amdgcn_readlane(v, 63);
  return max(max(a, b), max(c, d));
}
#define STAMP(idx) do {} while (0)

__global__ void __launch_bounds__(512, 2) fused_gemm_q8(Params P) {
  __shared__ __attribute__((aligned(1024))) char smem[LDS_BYTES];
  LAS char* lds = (LAS char*)smem;
  LAS unsigned* misc = (LAS unsigned*)(lds + 4 * PIECE);
  const int tid = threadIdx.x, lane = tid & 63, wv = __builtin_amdgcn_readfirstlane(tid >> 6);
  const int b = blockIdx.x, xg = b & 7, slot = b >> 3;
  const int rank = P.rank;
  char* myws = pick(P.ws, rank);
  char* tmo = myws + TMO_OFF;
  STAMP(0);
  char* cntp = myws + CNT_OFF + (size_t)b * 128;
  const unsigned cv = ld_agent(cntp);

  if (slot < 16) {
    const int rel = (slot + 1) & 7, unit = slot >> 3, dst = (rank + rel) & 7;
    char* dws = pick(P.ws, dst);
    const __amdgpu_buffer_rsrc_t rx = mkrsrc(P.x, MG * KSH * 4), rw = mkrsrc(P.w, (unsigned)KSH * NG * 4u);
    const __amdgpu_buffer_rsrc_t rd = mkrsrc(dws + (size_t)rank * SRC_BYTES, (unsigned)SRC_BYTES);
    char* dflag = dws + FLAG_OFF + (size_t)rank * 8 * 48 * 128;
    const int g = lane >> 4, c16 = lane & 15;
    const unsigned xrot = 8u * ((8 - rel) & 3);
    const int v = wv & 3;
    const unsigned xlane = (unsigned)(v * 32 + c16) * 2048u + g * 16u;
    const unsigned wlane = (unsigned)((v & 1) * 32 + 4 * g) * 65536u + (unsigned)(dst * NBLK + (v >> 1) * 64 + 4 * c16) * 4u;
#define LD128(dst, voff, rs, soff, imm) asm volatile("buffer_load_dwordx4 %0, %1, %2, %3 offen offset:" #imm : "=v"(dst) : "v"(voff), "s"(rs), "s"(soff) : "memory")
    auto issue = [&](f32x4 (&L)[8], unsigned q) {
      const unsigned c = q / 48u, e = q % 48u;
      if (e < 16u) {
        const unsigned so = __builtin_amdgcn_readfirstlane(c * 64u * 65536u + e * 512u);
        asm volatile("s_nop 4" :: "s"(so));
#pragma unroll
        for (int k = 0; k < 8; ++k) LD128(L[k], wlane, rw, so + (unsigned)(16 * (k >> 2) + (k & 3)) * 65536u, 0);
      } else {
        const unsigned hx = (e - 16u + xrot) & 31u;
        const unsigned so = __builtin_amdgcn_readfirstlane(hx * 128u * 2048u + c * 256u), so2 = so + 32768u;
        asm volatile("s_nop 4" :: "s"(so));
        LD128(L[0], xlane, rx, so, 0); LD128(L[1], xlane, rx, so, 64); LD128(L[2], xlane, rx, so, 128); LD128(L[3], xlane, rx, so, 192);
        LD128(L[4], xlane, rx, so2, 0); LD128(L[5], xlane, rx, so2, 64); LD128(L[6], xlane, rx, so2, 128); LD128(L[7], xlane, rx, so2, 192);
      }
    };
    f32x4 L[8];
    unsigned q0 = unit * 8 + xg, q1 = (UNITS + unit) * 8 + xg;
    const bool active = wv < 4 && unit < UNITS;
    if (active) issue(L, q0);
    LAS unsigned* tick = (LAS unsigned*)lds;
    LAS unsigned* cnt = tick + 512;
    if (tid < 256) *(LAS u32x4*)(lds + tid * 16) = (u32x4){0u, 0u, 0u, 0u};
    const unsigned call = __builtin_amdgcn_readfirstlane(cv) + 1;
    asm volatile("s_waitcnt lgkmcnt(0)" :: "s"(call) : "memory");
    BAR();
    if (wv == 7 && lane == 0) { st_agent(cntp, call); if (xg == 0 && unit == 0) st_agent(myws + TKT_OFF + (size_t)(((call + 2) & 3) * 8 + rel) * 128, 0u); }
    GAS unsigned* tkt = (GAS unsigned*)(myws + TKT_OFF + (size_t)((call & 3) * 8 + rel) * 128);
    if (active) {
      const unsigned stop = unit == 0 ? NJOB : (unsigned)TSTOP;
      u32x4 o[4]; unsigned doff, hp;
      auto convert = [&](unsigned q) {
        asm volatile("" : "+v"(L[0]), "+v"(L[1]), "+v"(L[2]), "+v"(L[3]), "+v"(L[4]), "+v"(L[5]), "+v"(L[6]), "+v"(L[7]));
        const unsigned c = q / 48u, e = q % 48u;
        if (e < 16u) {
#pragma unroll
          for (int t = 0; t < 4; ++t) o[t] = (u32x4){pk(L[0][t], L[1][t]), pk(L[2][t], L[3][t]), pk(L[4][t], L[5][t]), pk(L[6][t], L[7][t])};
          doff = 16u * PIECE + e * 16384u; hp = 32u + e;
        } else {
#pragma unroll
          for (int t = 0; t < 4; ++t) o[t] = (u32x4){pk(L[2 * t][0], L[2 * t][1]), pk(L[2 * t][2], L[2 * t][3]), pk(L[2 * t + 1][0], L[2 * t + 1][1]), pk(L[2 * t + 1][2], L[2 * t + 1][3])};
          const unsigned hx = (e - 16u + xrot) & 31u;
          doff = hx * 16384u; hp = hx;
        }
        doff += c * STEP_BYTES + (unsigned)v * 4096u + lane * 16u;
        hp += c * 48u;
        asm volatile("" : "+v"(o[0]), "+v"(o[1]), "+v"(o[2]), "+v"(o[3]), "+v"(doff));
      };
      WAIT_V0();
      convert(q0);
      for (int r = 0;; ++r) {
        unsigned drawn = NJOB;
        if (v == 0 && q1 < stop && lane == 0) {
          const unsigned one = 1u;
          asm volatile("global_atomic_add %0, %1, %2, off sc0" : "=v"(drawn) : "v"(tkt), "v"(one) : "memory");
        }
        if (q1 < NJOB) issue(L, q1);
#pragma unroll
        for (int t = 0; t < 4; ++t) __builtin_amdgcn_raw_buffer_store_b128(o[t], rd, doff + t * 1024u, 0, SYS);
        const unsigned cur_hp = hp;
        asm volatile("s_waitcnt vmcnt(4)" ::: "memory");
        unsigned q2 = NJOB;
        if (v == 0) {
          asm volatile("" : "+v"(drawn));
          q2 = __builtin_amdgcn_readfirstlane(drawn) + 2u * UNITS * 8u;
          if (lane == 0) tick[r] = q2 + 1u;
        }
        if (q1 < NJOB) convert(q1);
        if (v != 0) {
          unsigned tv;
          while ((tv = __builtin_amdgcn_readfirstlane(__hip_atomic_load(tick + r, __ATOMIC_RELAXED, __HIP_MEMORY_SCOPE_WORKGROUP))) == 0u) __builtin_amdgcn_s_sleep(1);
          q2 = tv - 1u;
        }
        WAIT_V0();
        unsigned old = 0;
        if (lane == 0) old = __hip_atomic_fetch_add(cnt + r, 1u, __ATOMIC_RELAXED, __HIP_MEMORY_SCOPE_WORKGROUP);
        if (__builtin_amdgcn_readfirstlane(old) == 3u && lane == 0) st_sys(dflag + (size_t)cur_hp * 128, call);
        if (q1 >= NJOB) break;
        q1 = __builtin_amdgcn_readfirstlane(q2);
      }
    }
    STAMP(1);
    if (b == 56 && wv == 0) {
      const unsigned long long t0 = __builtin_amdgcn_s_memrealtime();
      const char* dn0 = myws + DONE_OFF + lane * 4;
      for (unsigned it = 1;; ++it) {
        const unsigned a0 = ld_sys(dn0), a1 = ld_sys(dn0 + 256);
        const bool ok = (a0 >= call || (lane >> 4) == rank) && (a1 >= call || (lane >> 4) + 4 == rank);
        if (__all(ok)) break;
        if ((it & 63u) == 0u && give_up(tmo, call, t0)) { if (lane == 0) pick(P.out, rank)[rank * NBLK] = (char)-128; break; }
        __builtin_amdgcn_s_sleep(1);
      }
      STAMP(2);
    }
    return;
  }

  const int sl = slot - 16, own = sl >> 1, ti = 2 * own + (xg & 1), tj = 2 * (xg >> 1) + (sl & 1);
  const int wr = wv >> 2, wc = wv & 3;
  const unsigned call = __builtin_amdgcn_readfirstlane(cv) + 1;
  const int fsrc = (rank + 1 + ((lane >> 2) & 7)) & 7;
  const char* fl = myws + FLAG_OFF + (size_t)(fsrc * 8 * 48 + ((lane & 2) ? 32 + 2 * tj + (lane & 1) : 2 * ti + (lane & 1))) * 128;
  unsigned nxt = 0;
  const __amdgpu_buffer_rsrc_t rs = mkrsrc(myws, (unsigned)DATA_BYTES);
  const unsigned gx = (unsigned)ti * PIECE + wv * 4096 + lane * 16, gw = (unsigned)(16 + tj) * PIECE + wv * 4096 + lane * 16;
  constexpr unsigned NONE = 64;
#define BLDS(voff, dstoff, imm) __builtin_amdgcn_raw_ptr_buffer_load_lds(rs, (LAS void*)(lds + (dstoff)), 16, voff, so, imm, SYS)
#define STAGE(si, lb) do { const unsigned so = (si) * STEP_BYTES; \
    BLDS(gx, (lb), 0); BLDS(gx, (lb), 1024); BLDS(gx, (lb), 2048); BLDS(gx, (lb), 3072); \
    BLDS(gw, (lb) + PIECE, 0); BLDS(gw, (lb) + PIECE, 1024); BLDS(gw, (lb) + PIECE, 2048); BLDS(gw, (lb) + PIECE, 3072); } while (0)
#define RD(X, W, xa, wa, h) do { _Pragma("unroll") for (int t = 0; t < 4; ++t) W[t] = *(const LAS bf16x8*)(lds + (wa) + (h) * 4096 + t * 1024); \
    _Pragma("unroll") for (int m = 0; m < 8; ++m) X[m] = *(const LAS bf16x8*)(lds + (xa) + m * 2048 + (h) * 1024); } while (0)
#define MM(X, W) MMR(X, W, 0, 8)
#define MMR(X, W, m0, m1) do { _Pragma("unroll") for (int m = m0; m < m1; ++m) _Pragma("unroll") for (int t = 0; t < 4; ++t) \
    acc[m][t] = __builtin_amdgcn_mfma_f32_16x16x32_bf16(W[t], X[m], acc[m][t], 0, 0, 0); } while (0)
#define SGB(m, n) __builtin_amdgcn_sched_group_barrier(m, n, 0)
  bool dead = false;
  f32x4 acc[8][4];
#pragma unroll
  for (int m = 0; m < 8; ++m)
#pragma unroll
    for (int t = 0; t < 4; ++t) acc[m][t] = (f32x4){0.f, 0.f, 0.f, 0.f};
  bf16x8 X0[8], W0[4], X1[8], W1[4];
#pragma unroll
  for (int m = 0; m < 8; ++m) X0[m] = (bf16x8){0, 0, 0, 0, 0, 0, 0, 0};
#pragma unroll
  for (int t = 0; t < 4; ++t) W0[t] = (bf16x8){0, 0, 0, 0, 0, 0, 0, 0};
  unsigned xa = wr * 16384 + lane * 16, wa = PIECE + wc * 8192 + lane * 16;
  unsigned lb = wv * 4096;
  if (tid < 256) misc[tid] = 0;
  asm volatile("s_waitcnt lgkmcnt(0)" :: "s"(call) : "memory");
  BAR();
  if (tid == 0) st_agent(cntp, call);
  STAMP(1);
  unsigned empties = 0, sn = 4 * PIECE + 512;
  unsigned sna = sn + min(lane, 32) * 4;
  const unsigned long long tw = __builtin_amdgcn_s_memrealtime();
  auto decide = [&](unsigned fv) -> unsigned {
    const unsigned long long bal = __ballot(fv >= call);
    unsigned m = (unsigned)bal;
    m &= m >> 1; m &= m >> 2;
    m |= (unsigned)(bal >> 32);
    m &= 0x11111111u & ~(nxt >> 3);
    unsigned si = NONE;
    if (m != 0u) {
      const unsigned sh = __builtin_ctz(m);
      si = (unsigned)((rank + 1 + (sh >> 2)) & 7) * 8u + ((nxt >> sh) & 15u);
      nxt += 1u << sh;
    }
    return si;
  };
#define POLL() do { if (wv == 0 && lane < 32) \
      __builtin_amdgcn_global_load_lds((const GAS unsigned*)(fl + (size_t)min((nxt >> (lane & 28)) & 15u, 7u) * (48 * 128)), (LAS unsigned*)(lds + sn), 4, 0, SYS); } while (0)
  bool vcur = false, vnext = false;
  for (int done = 0; done < 64; ++done) {
    if (!vcur) {
      bool had;
      do {
        POLL();
        if (wv == 0 && !vnext && (dead || ((++empties & 63u) == 0u && give_up(tmo, call, tw)))) {
          dead = true;
          if (lane == 0) *(LAS unsigned*)(lds + sn + 128) = ~0u;
        }
        asm volatile("s_waitcnt vmcnt(0) lgkmcnt(0)" ::: "memory");
        BAR();
        const unsigned si = decide(*(const LAS unsigned*)(lds + sna));
        if (si != NONE) STAGE(si, lb);
        xa ^= 2 * PIECE; wa ^= 2 * PIECE; lb ^= 2 * PIECE; sn ^= 256u; sna ^= 256u;
        had = vnext; vnext = si != NONE;
      } while (!had);
      RD(X0, W0, xa, wa, 0);
    }
    POLL();
    RD(X1, W1, xa, wa, 1);
    MM(X0, W0);
#pragma unroll
    for (int k = 0; k < 12; ++k) { SGB(0x008, 2); SGB(0x100, 1); }
    SGB(0x008, 8);
    __builtin_amdgcn_sched_barrier(0);
    asm volatile("s_waitcnt vmcnt(0) lgkmcnt(0)" ::: "memory");
    BAR();
    __builtin_amdgcn_sched_barrier(0);
    const unsigned fv = *(const LAS unsigned*)(lds + sna);
    xa ^= 2 * PIECE; wa ^= 2 * PIECE;
    MMR(X1, W1, 0, 2);
    __builtin_amdgcn_sched_barrier(0);
    const unsigned si = decide(fv);
    STAGE(si, lb);
    RD(X0, W0, xa, wa, 0);
    MMR(X1, W1, 2, 8);
#pragma unroll
    for (int k = 0; k < 8; ++k) { SGB(0x020, 1); SGB(0x008, 1); }
#pragma unroll
    for (int k = 0; k < 12; ++k) { SGB(0x100, 1); SGB(0x008, 1); }
    SGB(0x008, 4);
    __builtin_amdgcn_sched_barrier(0);
    lb ^= 2 * PIECE; sn ^= 256u; sna ^= 256u;
    vcur = vnext; vnext = si != NONE;
  }
  STAMP(2);
  float mx = 0.f;
#pragma unroll
  for (int m = 0; m < 8; ++m)
#pragma unroll
    for (int t = 0; t < 4; ++t) { mx = __builtin_fmaxf(__builtin_fmaxf(mx, acc[m][t][0]), acc[m][t][1]); mx = __builtin_fmaxf(__builtin_fmaxf(mx, acc[m][t][2]), acc[m][t][3]); }
  {
    const unsigned wm = wave_umax(__float_as_uint(mx));
    unsigned old = 0;
    if (lane == 0) {
      __hip_atomic_fetch_max(misc + 0, wm, __ATOMIC_RELAXED, __HIP_MEMORY_SCOPE_WORKGROUP);
      old = __hip_atomic_fetch_add(misc + 1, 1u, __ATOMIC_RELAXED, __HIP_MEMORY_SCOPE_WORKGROUP);
    }
    if (__builtin_amdgcn_readfirstlane(old) == 7u) {
      const unsigned mb = __hip_atomic_load(misc + 0, __ATOMIC_RELAXED, __HIP_MEMORY_SCOPE_WORKGROUP);
      if (lane < NR) {
        char* gp = P.ws[0];
#pragma unroll
        for (int k = 1; k < NR; ++k) gp = (lane == k) ? P.ws[k] : gp;
        __hip_atomic_store((GAS unsigned long long*)(gp + GRAN_OFF + (size_t)(rank * 128 + ti * 8 + tj) * 8), ((unsigned long long)call << 32) | mb,
                           __ATOMIC_RELAXED, __HIP_MEMORY_SCOPE_SYSTEM);
      }
    }
  }
  {
    const __amdgpu_buffer_rsrc_t rg = mkrsrc(myws + GRAN_OFF, 8192);
    const unsigned long long t0 = __builtin_amdgcn_s_memrealtime();
    unsigned mb = 0;
    for (unsigned it = 1;; ++it) {
      const u32x4 gq = __builtin_amdgcn_raw_buffer_load_b128(rg, wv * 1024 + lane * 16, 0, SYS);
      const unsigned tv = ld_agent(tmo);
      mb = max(gq.x, gq.z);
      dead = dead || __builtin_amdgcn_readfirstlane(tv) == call;
      if (dead || __all(gq.y == call && gq.w == call)) break;
      if ((it & 63u) == 0u && give_up(tmo, call, t0)) { dead = true; break; }
      __builtin_amdgcn_s_sleep(1);
      asm volatile("" ::: "memory");
    }
    mb = wave_umax(mb);
    if (lane == 0) { __hip_atomic_fetch_max(misc + 2, mb, __ATOMIC_RELAXED, __HIP_MEMORY_SCOPE_WORKGROUP); if (dead) misc[3] = 1u; }
  }
  WAIT_L0();
  BAR();
  STAMP(3);
  const float amax = __uint_as_float(misc[2]);
  const bool bad = misc[3] != 0u;
  const float scale = amax / 127.0f;
  const float inv = scale > 0.f ? 1.0f / scale : 0.f;
  const __amdgpu_buffer_rsrc_t ro = mkrsrc(pick(P.out, own), (unsigned)MSH * NG);
  const unsigned obase = (unsigned)((ti & 1) * 256 + wr * 128 + (lane & 15)) * (unsigned)NG + (unsigned)(rank * NBLK + tj * 256 + wc * 64 + (lane >> 4) * 16);
#pragma unroll
  for (int m = 0; m < 8; ++m) {
    u32x4 q;
#pragma unroll
    for (int j = 0; j < 4; ++j) {
      unsigned by[4];
#pragma unroll
      for (int t = 0; t < 4; ++t) by[t] = __float_as_uint(__builtin_amdgcn_fmed3f(acc[m][t][j] * inv, 0.f, 127.0f) + 12582912.0f);
      const unsigned lo = __builtin_amdgcn_perm(by[1], by[0], 0x0c0c0400u), hi = __builtin_amdgcn_perm(by[3], by[2], 0x04000c0cu);
      q[j] = bad ? 0x80808080u : (lo | hi);
    }
    __builtin_amdgcn_raw_buffer_store_b128(q, ro, obase + (unsigned)m * 16u * NG, 0, SYS);
  }
  WAIT_V0();
  {
    unsigned old = 0;
    if (lane == 0) old = __hip_atomic_fetch_add(misc + 4, 1u, __ATOMIC_RELAXED, __HIP_MEMORY_SCOPE_WORKGROUP);
    if (__builtin_amdgcn_readfirstlane(old) == 7u && lane == 0 && own != rank)
      st_sys(pick(P.ws, own) + DONE_OFF + (size_t)(rank * 16 + (ti & 1) * 8 + tj) * 4, call);
  }
  STAMP(4);
}

}

extern "C" void kernel_launch(int rank, int nranks, ncclComm_t comm, hipStream_t stream,
                              void* const* d_in, const int* in_sizes, int n_in,
                              void* d_out, int out_size,
                              const int* in_dtypes, int out_dtype,
                              void* const* d_out_all,
                              void* const* d_ws_all, size_t ws_bytes) {
  if (nranks != NR || (unsigned)rank >= (unsigned)NR || n_in != 2 || in_sizes[0] != MG * KSH || in_sizes[1] != KSH * NG ||
      out_size != MSH * NG || ws_bytes < WS_NEED) {
    fprintf(stderr, "unexpected problem shape\n");
    return;
  }
  Params P;
  P.x = (const float*)d_in[0]; P.w = (const float*)d_in[1]; P.rank = rank;
  for (int p = 0; p < NR; ++p) { P.ws[p] = (char*)d_ws_all[p]; P.out[p] = (char*)d_out_all[p]; }
  hipLaunchKernelGGL(fused_gemm_q8, dim3(NWG), dim3(512), 0, stream, P);
  HIP_CHECK(hipGetLastError());
}
```

```cpp
#include <hip/hip_runtime.h>
#include <rccl/rccl.h>
#include <cstdio>

#define HIP_CHECK(expr) do { hipError_t _e = (expr); if (_e != hipSuccess) { \
    fprintf(stderr, "HIP error %s at %s:%d\n", hipGetErrorString(_e), __FILE__, __LINE__); \
    return; } } while (0)

namespace {

typedef short bf16x8 __attribute__((ext_vector_type(8)));
typedef __bf16 bf16x2_t __attribute__((ext_vector_type(2)));
typedef float f32x4 __attribute__((ext_vector_type(4)));
typedef float f32x2 __attribute__((ext_vector_type(2)));
typedef unsigned u32x4 __attribute__((ext_vector_type(4)));
typedef unsigned u32x2 __attribute__((ext_vector_type(2)));
typedef unsigned long long u64;
typedef __amdgpu_buffer_rsrc_t rsrc_t;
#define LAS __attribute__((address_space(3)))
#define GAS __attribute__((address_space(1)))

constexpr int NR = 8;
constexpr int MG = 4096, KSH = 512, NG = 16384, NBLK = 2048, MSH = 512;
constexpr unsigned PIECE = 16384;
constexpr unsigned UNIT_BYTES = 24 * PIECE;
constexpr unsigned SRC_BYTES = 16 * UNIT_BYTES;
constexpr unsigned DATA_BYTES = NR * SRC_BYTES;
constexpr unsigned FLAG_SRC = 18 * 24 * 128;
constexpr unsigned FLAG_OFF = DATA_BYTES;
constexpr unsigned RDY_OFF = FLAG_OFF + NR * FLAG_SRC;
constexpr unsigned LMAX_OFF = RDY_OFF + NR * FLAG_SRC;
constexpr unsigned GMAX_OFF = LMAX_OFF + 1024;
constexpr unsigned TMO_OFF = GMAX_OFF + NR * 128;
constexpr unsigned UCNT_OFF = TMO_OFF + 256;
constexpr unsigned TAM_OFF = TMO_OFF + 128;
constexpr unsigned MFLG_OFF = TMO_OFF + 4096;
constexpr unsigned MASK_BLK = 1024, QBLK = 9216;
constexpr unsigned MASK_OFF = MFLG_OFF + NR * 16 * 128;
constexpr unsigned QDAT_OFF = MASK_OFF + NR * 128 * MASK_BLK;
constexpr unsigned CNT_OFF = QDAT_OFF + NR * 128 * QBLK;
constexpr unsigned TKT_OFF = CNT_OFF + 256 * 128;
constexpr unsigned WEXP_OFF = 64u << 20;
constexpr unsigned WS_NEED = WEXP_OFF + NR * 128 * PIECE;
static_assert(TKT_OFF + 4096 <= WEXP_OFF && UCNT_OFF + 16 * 128 <= MFLG_OFF, "layout");
constexpr int NCONV = 56, NPUSH = 112, NGEMM = 128, NWG = NPUSH + NGEMM + 1;
constexpr u64 DEADLINE = 50000000ull;
constexpr int SYS = 17, DEV = 16;
constexpr unsigned BUF = 4 * PIECE;
constexpr unsigned LUT = 2 * BUF + 1024;
constexpr int LDS_BYTES = 2 * BUF + 2048;
constexpr unsigned NJOB = 384;
constexpr unsigned LOOK = 2;
constexpr unsigned UF = 2;
constexpr unsigned PFU = 13;
constexpr int CWN = 3;
constexpr int MD = 4;
constexpr unsigned ODLY = 70, OSL = 250, OBW = 42;

struct Params {
  const float* x; const float* w;
  char* out; int rank, pad;
  char* ws[NR];
};

__device__ __forceinline__ u64 uni64(u64 a) {
  const unsigned lo = __builtin_amdgcn_readfirstlane((unsigned)a), hi = __builtin_amdgcn_readfirstlane((unsigned)(a >> 32));
  return ((u64)hi << 32) | lo;
}
__device__ __forceinline__ rsrc_t mkrsrc(const void* p, unsigned bytes) {
  return __builtin_amdgcn_make_buffer_rsrc((void*)uni64((u64)p), 0, bytes, 0x00020000);
}
__device__ __forceinline__ unsigned pk(float a, float b) {
  return __builtin_bit_cast(unsigned, __builtin_convertvector((f32x2){a, b}, bf16x2_t));
}
__device__ __forceinline__ unsigned ld_sys(const char* p) { return __hip_atomic_load((const GAS unsigned*)p, __ATOMIC_RELAXED, __HIP_MEMORY_SCOPE_SYSTEM); }
__device__ __forceinline__ void st_sys(char* p, unsigned v) { __hip_atomic_store((GAS unsigned*)p, v, __ATOMIC_RELAXED, __HIP_MEMORY_SCOPE_SYSTEM); }
__device__ __forceinline__ unsigned ld_agent(const char* p) { return __hip_atomic_load((const GAS unsigned*)p, __ATOMIC_RELAXED, __HIP_MEMORY_SCOPE_AGENT); }
__device__ __forceinline__ void st_agent(char* p, unsigned v) { __hip_atomic_store((GAS unsigned*)p, v, __ATOMIC_RELAXED, __HIP_MEMORY_SCOPE_AGENT); }
__device__ __forceinline__ unsigned lds_get(LAS unsigned* p) { return __builtin_amdgcn_readfirstlane(__hip_atomic_load(p, __ATOMIC_RELAXED, __HIP_MEMORY_SCOPE_WORKGROUP)); }
__device__ __forceinline__ unsigned lds_arrive(LAS unsigned* p, int lane) {
  unsigned old = 0;
  if (lane == 0) old = __hip_atomic_fetch_add(p, 1u, __ATOMIC_RELAXED, __HIP_MEMORY_SCOPE_WORKGROUP);
  return __builtin_amdgcn_readfirstlane(old);
}
#define WAIT_V(n) asm volatile("s_waitcnt vmcnt(" #n ")" ::: "memory")
#define WAIT_L0() asm volatile("s_waitcnt lgkmcnt(0)" ::: "memory")
#define WAIT_ALL() asm volatile("s_waitcnt vmcnt(0) lgkmcnt(0)" ::: "memory")
#define BAR() do { asm volatile("" ::: "memory"); __builtin_amdgcn_s_barrier(); asm volatile("" ::: "memory"); } while (0)
#define ST128(v, rs, off, aux) __builtin_amdgcn_raw_buffer_store_b128(v, rs, off, 0, aux)
#define LD128S(rs, off) __builtin_amdgcn_raw_buffer_load_b128(rs, off, 0, SYS)
__device__ __forceinline__ bool give_up(char* tmo, unsigned call, u64 t0) {
  const bool dead = __builtin_amdgcn_readfirstlane(ld_agent(tmo)) == call || __builtin_amdgcn_s_memrealtime() - t0 > DEADLINE;
  if (dead) st_agent(tmo, call);
  return dead;
}
#define DPP(v, ctrl, rows) ((unsigned)__builtin_amdgcn_update_dpp(0, (int)(v), ctrl, rows, 0xf, true))
__device__ __forceinline__ unsigned wave_umax(unsigned v) {
  v = max(v, DPP(v, 0x111, 0xf)); v = max(v, DPP(v, 0x112, 0xf)); v = max(v, DPP(v, 0x114, 0xf)); v = max(v, DPP(v, 0x118, 0xf));
  return max(max((unsigned)__builtin_amdgcn_readlane(v, 15), (unsigned)__builtin_amdgcn_readlane(v, 31)),
             max((unsigned)__builtin_amdgcn_readlane(v, 47), (unsigned)__builtin_amdgcn_readlane(v, 63)));
}
__device__ __forceinline__ unsigned wave_scan(unsigned v) {
  v += DPP(v, 0x111, 0xf); v += DPP(v, 0x112, 0xf); v += DPP(v, 0x114, 0xf); v += DPP(v, 0x118, 0xf);
  v += DPP(v, 0x142, 0xa); v += DPP(v, 0x143, 0xc);
  return v;
}
__device__ __forceinline__ int lane_id() {
  int l = __builtin_amdgcn_mbcnt_hi(~0u, __builtin_amdgcn_mbcnt_lo(~0u, 0u));
  asm volatile("" : "+v"(l));
  return l;
}
__device__ __forceinline__ u32x4 call_tag(unsigned call) {
  const unsigned t[4] = {call % 15u + 1u, call % 16u, call % 13u, call % 11u};
  u32x4 r;
#pragma unroll
  for (int d = 0; d < 4; ++d) r[d] = ((t[d] & 1u) << 7) | ((t[d] & 2u) << 14) | ((t[d] & 4u) << 21) | ((t[d] & 8u) << 28);
  return r;
}

__global__ void __launch_bounds__(512, 2) fused_gemm_q8(Params P) {
  __shared__ __attribute__((aligned(1024))) char smem[LDS_BYTES];
  LAS char* lds = (LAS char*)smem;
  LAS unsigned* misc = (LAS unsigned*)(lds + 2 * BUF);
  const unsigned xg = blockIdx.x & 7, ps = blockIdx.x >> 3;
  typedef unsigned u32x8 __attribute__((ext_vector_type(8)));
  u32x8 ka; u64 kw0, kwp;
  asm volatile("s_load_dwordx8 %0, %3, 0x0\n\ts_load_dwordx2 %1, %3, 0x20\n\ts_load_dwordx2 %2, %3, %4 offset:0x20"
               : "=&s"(ka), "=&s"(kw0), "=&s"(kwp) : "s"(__builtin_amdgcn_kernarg_segment_ptr()), "s"(ps < 7 ? ps * 8 + 8 : 0u) : "memory");
  static_assert(offsetof(Params, w) == 8 && offsetof(Params, rank) == 24 && offsetof(Params, ws) == 32, "argument layout");
  const int wv = __builtin_amdgcn_readfirstlane(threadIdx.x >> 6);
  const int slot = ps < 7 ? ps + 14 : ps < 21 ? ps - 7 : ps, b = slot * 8 + xg;
  const bool conv = b < NCONV, isg = b >= NPUSH && b < NPUSH + NGEMM;
  unsigned call, rank;
  char* myws;
  {
    const int lane = lane_id(), tid = wv * 64 + lane;
    const bool active = ps < 14 && wv < 4;
    constexpr unsigned NCJ = 24 + 15 * 80;
    unsigned ja = xg * 8u + (ps < 7 ? ps + 1u : 0u), jb = b + (ps < 9 ? 64u : 104u);
    asm volatile("" : "+s"(ja), "+s"(jb));
    unsigned n = conv ? b : NCJ, jc = (ps < 7 || ps == 9) ? ja : jb;
    const unsigned xlane = (unsigned)(wv * 64 + (lane >> 2)) * 2048u + (unsigned)((lane & 3) ^ (-(lane >> 4) & 3)) * 32u;
    const unsigned wlane = (unsigned)(8 * (lane >> 4)) * 65536u + (unsigned)(wv * 64 + 4 * (lane & 15)) * 4u;
    asm volatile("s_waitcnt lgkmcnt(0)" : "+s"(ka), "+s"(kw0), "+s"(kwp) : "v"(xlane), "v"(wlane), "s"(jc) : "memory");
    rank = ka[6]; myws = (char*)kw0;
    u64 px = ((u64)ka[1] << 32) | ka[0], pw = ((u64)ka[3] << 32) | ka[2];
    asm volatile("" : "+s"(px), "+s"(pw));
    f32x4 L[8];
    auto issue = [&](unsigned jc) {
      const unsigned u = jc >> 7, e = jc & 127u;
      const bool isw = e < 64u;
      unsigned sw = u * 32u * 65536u + ((rank + e) & 7u) * (NBLK * 4u) + (e >> 3) * 1024u, sx = (e - 64u) * 256u * 2048u + u * 128u;
      asm volatile("" : "+s"(sw), "+s"(sx));
      const rsrc_t rin = mkrsrc((const void*)(isw ? pw : px), isw ? (unsigned)KSH * NG * 4u : (unsigned)MG * KSH * 4u);
      const unsigned so = isw ? sw : sx, sa = isw ? 131072u : 32768u, sb = isw ? 65536u : 16u, vo = isw ? wlane : xlane;
      asm volatile("s_nop 4" :: "s"(rin), "s"(so));
#pragma unroll
      for (int k = 0; k < 8; ++k)
        asm volatile("buffer_load_dwordx4 %0, %1, %2, %3 offen" : "=v"(L[k]) : "v"(vo), "s"(rin), "s"(so + (k >> 1) * sa + (k & 1) * sb) : "memory");
    };
    if (active) issue(jc);
    char* cntp = myws + CNT_OFF + b * 128;
    unsigned cv; char* fb;
    asm volatile("global_load_dword %0, %1, off sc1" : "=v"(cv) : "v"(cntp) : "memory");
    asm volatile("global_load_dwordx2 %0, %1, off" : "=v"(fb) : "v"((const char*)__builtin_amdgcn_kernarg_segment_ptr() + offsetof(Params, ws) + (lane & 7) * 8) : "memory");
    const rsrc_t rp = mkrsrc((const void*)kwp, WS_NEED);
    for (int r = 0;; ++r) {
      const unsigned u = jc >> 7, e = jc & 127u, pc = u * 24u + (e < 64u ? 16u + (e >> 3) : e - 64u);
      if (active) {
        unsigned de = WEXP_OFF + (((rank + e) & 7u) * 128u + u * 8u + (e >> 3)) * PIECE, dd = rank * SRC_BYTES + pc * PIECE;
        asm volatile("" : "+s"(de), "+s"(dd));
        const unsigned doff = ((conv && e < 64u && (e & 7u) != 0u) ? de : dd) + wv * 4096u + lane * 16u;
        if (r == 0) asm volatile("s_waitcnt vmcnt(2)" :: "v"(doff) : "memory");
        asm volatile("" : "+v"(L[0]), "+v"(L[1]), "+v"(L[2]), "+v"(L[3]), "+v"(L[4]), "+v"(L[5]), "+v"(L[6]), "+v"(L[7]));
        if (e < 64u) {
#pragma unroll
          for (int t = 0; t < 4; ++t) ST128(((u32x4){pk(L[0][t], L[1][t]), pk(L[2][t], L[3][t]), pk(L[4][t], L[5][t]), pk(L[6][t], L[7][t])}), rp, doff + t * 1024u, SYS);
        } else {
#pragma unroll
          for (int t = 0; t < 4; ++t)
            ST128(((u32x4){pk(L[2 * t][0], L[2 * t][1]), pk(L[2 * t][2], L[2 * t][3]), pk(L[2 * t + 1][0], L[2 * t + 1][1]), pk(L[2 * t + 1][2], L[2 * t + 1][3])}), rp, doff + t * 1024u, SYS);
        }
      }
      if (r == 0) {
        if (tid >= 256) { *(LAS u32x4*)(lds + (tid - 256) * 16) = (u32x4){0u, 0u, 0u, 0u}; misc[tid - 256] = 0u; }
        if (wv == 7) {
          if (isg) {
            u64 st = 0x0c0c0c0c0c0c0c0cull;
            unsigned at = 0;
#pragma unroll
            for (int t = 0; t < 3; ++t) if (t < (lane >> 4)) { st = (st & ~(0xffull << (8 * at))) | ((u64)t << (8 * at)); ++at; }
#pragma unroll
            for (int t = 0; t < 4; ++t) if ((lane >> t) & 1) { st = (st & ~(0xffull << (8 * at))) | ((u64)(4 + t) << (8 * at)); ++at; }
            *(LAS u32x2*)(lds + LUT + lane * 8) = (u32x2){(unsigned)st, at >= 4 ? (unsigned)(st >> 32) : (unsigned)st};
          } else {
            unsigned sel = 0, at = lane >> 4;
#pragma unroll
            for (int t = 0; t < 4; ++t) { const bool on = (lane >> t) & 1; sel |= (on ? at : 12u) << (8 * t); at += on; }
            *(LAS unsigned*)(lds + LUT + lane * 4) = sel;
          }
        }
        if (active) WAIT_V(4); else WAIT_V(0);
        asm volatile("" : "+v"(cv), "+v"(fb));
        call = __builtin_amdgcn_readfirstlane(cv) + 1;
        fb += ((isg || lane == 0) ? FLAG_OFF : RDY_OFF) + rank * FLAG_SRC;
        asm volatile("s_waitcnt lgkmcnt(0)" :: "s"(call) : "memory");
        BAR();
        if (wv == 7 && lane == 0) { st_agent(cntp, call); if (b < 7) st_agent(myws + TKT_OFF + (((call + 2) & 3) * 8 + 1 + b) * 128, 0u); }
      }
      if (!active) break;
      const unsigned fm = e < 64u ? 1u << (e & 7u) : 0xffu;
      n += NCONV;
      if (n < NCJ) { jc = ((n + 56u) / 80u) * 128u + (n + 56u) % 80u; issue(jc); WAIT_V(8); } else WAIT_V(0);
      const bool last = lds_arrive(misc + 64 + r, lane) == 3u;
      bool sent = false;
      if (last) {
        unsigned m = u < UF ? fm : fm & 1u;
        if (u >= UF) {
          unsigned k = 0;
          if (lane == 0) k = __hip_atomic_fetch_add((GAS unsigned*)(myws + UCNT_OFF + u * 128u), 1u, __ATOMIC_RELAXED, __HIP_MEMORY_SCOPE_AGENT);
          if ((__builtin_amdgcn_readfirstlane(k) + 1u) % 80u == 0u) m |= 0xfeu;
        }
        if (((m >> lane) & 1u) && lane < NR) st_sys(fb + (u < UF || lane == 0 ? pc : u * 24u) * 128u, call);
        sent = m != 0u;
      }
      if (n >= NCJ) break;
      if (sent) WAIT_V(1); else WAIT_V(0);
    }
  }
  char* tmo = myws + TMO_OFF;

  if (!isg) {
    const int lane = lane_id();
    const u64 t0 = __builtin_amdgcn_s_memrealtime();
    if (b == NWG - 1) {
      if (wv < CWN) {
        const rsrc_t rl = mkrsrc(myws + LMAX_OFF, 1024);
        char* gp = P.ws[lane & 7] + GMAX_OFF + rank * 128;
        const unsigned ma = (unsigned)(size_t)misc;
        for (unsigned it = 1;; ++it) {
          const u32x4 g = LD128S(rl, lane * 16);
          u64 note = ((u64)call << 32) | wave_umax(max(g.x, g.z));
          const bool all = __all(g.y == call && g.w == call);
          unsigned old = 1;
          if (lane == 0) asm volatile("ds_add_rtn_u32 %0, %1, %2\n\ts_waitcnt lgkmcnt(0)" : "=&v"(old) : "v"(ma), "v"(all ? 1u : 0u) : "memory");
          unsigned send = all && __builtin_amdgcn_readfirstlane(old) == 0u ? NR : 0;
          asm volatile("" : "+v"(note), "+v"(gp), "+s"(send));
          if (lane < send) __hip_atomic_store((GAS u64*)gp, note, __ATOMIC_RELAXED, __HIP_MEMORY_SCOPE_SYSTEM);
          asm volatile("" ::: "memory");
          if (all || ((it & 63u) == 0u && give_up(tmo, call, t0))) break;
        }
      } else if (wv == CWN) {
        const char* gp = myws + GMAX_OFF + (lane & 7) * 128 + 4;
        for (unsigned it = 1;; ++it) {
          const bool all = __all(ld_sys(gp) == call) || ((it & 63u) == 0u && give_up(tmo, call, t0));
          u64 note = ((u64)call << 32) | (unsigned)__builtin_amdgcn_s_memrealtime();
          asm volatile("" : "+v"(note));
          if (all && lane == 0) __hip_atomic_store((GAS u64*)(myws + TAM_OFF), note, __ATOMIC_RELAXED, __HIP_MEMORY_SCOPE_AGENT);
          if (all) break;
        }
      }
      return;
    }
    if (!conv) {
      const unsigned rel = slot - 6, peer = (rank + rel) & 7u, xrot = (rel - 1u) * 16u / 7u;
      auto piece = [&](unsigned q) { const unsigned u = q / 24u, e = q % 24u; return u * 24u + (e < 8u ? 16u + e : (e - 8u + xrot) & 15u); };
      LAS unsigned* tick = (LAS unsigned*)lds;
      LAS unsigned* cnt = tick + 512;
      LAS unsigned* prog = cnt + 511;
      if (wv == 0) {
        GAS unsigned* tkt = (GAS unsigned*)(myws + TKT_OFF + ((call & 3) * 8 + rel) * 128);
        unsigned known = 0;
        for (unsigned r = 0;; ++r) {
          unsigned q = 8u + 8u * r + xg;
          if (r >= 2u) {
            while (lds_get(prog) + LOOK < r) __builtin_amdgcn_s_sleep(2);
            unsigned d = 0;
            if (lane == 0) d = __hip_atomic_fetch_add(tkt, 1u, __ATOMIC_RELAXED, __HIP_MEMORY_SCOPE_AGENT);
            q = __builtin_amdgcn_readfirstlane(d) + 24u;
          }
          unsigned pc = q < NJOB ? piece(q) : NJOB;
          for (unsigned it = 1; pc < NJOB && !((known >> (pc / 24u)) & 1u); ++it) {
            const unsigned m = (unsigned)__ballot(ld_sys(myws + RDY_OFF + peer * FLAG_SRC + ((lane & 15) < UF ? pc : (lane & 15) * 24u) * 128u) >= call);
            known = m & (0xffffu << UF) & 0xffffu;
            if (pc < UF * 24u && (m & 1u)) break;
            if ((it & 63u) == 0u && give_up(tmo, call, t0)) pc = NJOB;
          }
          if (lane == 0) tick[r] = pc + 1u;
          if (pc >= NJOB) break;
        }
      } else if (wv >= 4) {
        char* pws = P.ws[rel & 7u];
        const rsrc_t rp = mkrsrc(pws, WS_NEED), rl = mkrsrc(myws + peer * SRC_BYTES, SRC_BYTES);
        char* lf = myws + FLAG_OFF + peer * FLAG_SRC;
        const unsigned lo = (unsigned)(wv & 3) * 4096u + lane * 16u;
        auto next = [&](unsigned r) -> unsigned {
          unsigned tv;
          while ((tv = lds_get(tick + r)) == 0u) __builtin_amdgcn_s_sleep(1);
          return tv - 1u;
        };
        auto srcoff = [&](unsigned pc) -> unsigned {
          const unsigned u = pc / 24u, t = pc % 24u;
          return __builtin_amdgcn_readfirstlane(t < 16u ? peer * SRC_BYTES + pc * PIECE : WEXP_OFF + (rank * 128u + u * 8u + t - 16u) * PIECE);
        };
#define PULL2(A, B, rs, so, i0, i1) asm volatile("s_nop 4\n\tbuffer_load_dwordx4 %0, %2, %3, %4 offen offset:" #i0 " sc0 sc1\n\t" \
          "buffer_load_dwordx4 %1, %2, %3, %4 offen offset:" #i1 " sc0 sc1" : "=&v"(A), "=&v"(B) : "v"(lo), "s"(rs), "s"(so) : "memory")
        u32x4 A0, A1, B0, B1;
        unsigned pc = next(0);
        if (pc < NJOB) {
          unsigned so = srcoff(pc);
          PULL2(A0, A1, rp, so, 0, 1024);
          bool sent = false;
          for (unsigned r = 0;; ++r) {
            const unsigned dof = pc * PIECE + lo;
            if (sent) WAIT_V(1); else WAIT_V(0);
            asm volatile("" : "+v"(A0), "+v"(A1));
            PULL2(B0, B1, rp, so, 2048, 3072);
            ST128(A0, rl, dof, DEV); ST128(A1, rl, dof + 1024u, DEV);
            const unsigned pn = next(r + 1u);
            so = srcoff(min(pn, NJOB - 1u));
            const rsrc_t rn = mkrsrc(pws, pn < NJOB ? WS_NEED : 0u);
            asm volatile("s_waitcnt vmcnt(0)" : "+v"(B0), "+v"(B1) :: "memory");
            ST128(B0, rl, dof + 2048u, DEV); ST128(B1, rl, dof + 3072u, DEV);
            PULL2(A0, A1, rn, so, 0, 1024);
            WAIT_V(2);
            sent = lds_arrive(cnt + r, lane) == 3u;
            if (sent && lane == 0) { st_agent(lf + pc * 128u, call); __hip_atomic_fetch_max(prog, r + 1u, __ATOMIC_RELAXED, __HIP_MEMORY_SCOPE_WORKGROUP); }
            if (pn >= NJOB) break;
            pc = pn;
          }
          WAIT_V(0);
        }
      }
    }
    if (conv) {
      const unsigned sp = (rank + 1 + slot) & 7u;
      const rsrc_t rd = mkrsrc(myws + sp * SRC_BYTES, SRC_BYTES);
      for (unsigned k = 2 * PFU; k < 32u; ++k) {
        if ((k & 1u) == 0u && wv >= 2) continue;
        const unsigned pc = (k >> 1) * 24u + ((k & 1u) ? 2 * wv + (xg & 1) : 16 + 2 * (xg >> 1) + wv);
        bool quit = false;
        for (unsigned it = 1; !quit; ++it) {
          if (__builtin_amdgcn_readfirstlane(ld_sys(myws + FLAG_OFF + sp * FLAG_SRC + pc * 128u)) >= call) break;
          if ((it & 63u) == 0u && give_up(tmo, call, t0)) quit = true;
          if (k < 2 * PFU + 2) __builtin_amdgcn_s_sleep(16);
        }
        if (quit) break;
        asm volatile("" ::: "memory");
        const unsigned p0 = __builtin_amdgcn_raw_buffer_load_b32(rd, pc * PIECE + lane * 128, 0, SYS);
        const unsigned p1 = __builtin_amdgcn_raw_buffer_load_b32(rd, pc * PIECE + 8192 + lane * 128, 0, SYS);
        asm volatile("" :: "v"(p0), "v"(p1));
      }
    }
    BAR();
    const unsigned srel = 1 + (b >> 4), src = (rank + srel) & 7u, ts = b & 15;
    char* sws = P.ws[srel & 7u];
    const unsigned blk = rank * 128 + ts * 8 + wv;
    bool dead = false;
    if (wv == 7) {
      for (unsigned it = 1; !dead; ++it) {
        const u64 g = __hip_atomic_load((const GAS u64*)(sws + MFLG_OFF + (rank * 16 + ts) * 128 + (lane & 7) * 8), __ATOMIC_RELAXED, __HIP_MEMORY_SCOPE_SYSTEM);
        const bool cur = (unsigned)(g >> 32) == call;
        if (cur && lane < 8) misc[8 + lane] = (unsigned)g;
        if (__all(cur)) break;
        if ((it & 63u) == 0u && give_up(tmo, call, t0)) dead = true;
      }
      if (dead && lane < 8) misc[8 + lane] = 1u;
    }
    unsigned tot;
    while ((tot = lds_get(misc + 8 + wv)) == 0u) __builtin_amdgcn_s_sleep(2);
    dead = tot == 1u;
    tot -= 2u;
    for (int k = 0; k < wv; ++k) __builtin_amdgcn_s_sleep(MD);
    const rsrc_t rq = mkrsrc(sws + QDAT_OFF + blk * QBLK, min((tot + 15u) & ~15u, QBLK));
    const unsigned slot_at = wv == 0 ? ODLY : OSL + wv * tot / OBW;
    u64 tam = __hip_atomic_load((const GAS u64*)(myws + TAM_OFF), __ATOMIC_RELAXED, __HIP_MEMORY_SCOPE_AGENT);
    auto due = [&]() { return (unsigned)__builtin_amdgcn_s_memrealtime() - (unsigned)tam >= slot_at || (wv != 0 && lds_get(misc) >= (unsigned)wv); };
    tam = uni64(tam);
    const bool early = (unsigned)(tam >> 32) == call && due();
    u32x4 mk, v[5];
    {
      const rsrc_t rm = mkrsrc(sws + MASK_OFF + blk * MASK_BLK, MASK_BLK), re = mkrsrc(sws + QDAT_OFF + blk * QBLK, early ? min((tot + 15u) & ~15u, QBLK) : 0u);
      const unsigned lo = lane * 16, hi = lo + 4096;
      asm volatile("s_nop 4\n\tbuffer_load_dwordx4 %0, %6, %8, 0 offen sc0 sc1\n\tbuffer_load_dwordx4 %1, %6, %9, 0 offen sc0 sc1\n\t"
                   "buffer_load_dwordx4 %2, %6, %9, 0 offen offset:1024 sc0 sc1\n\tbuffer_load_dwordx4 %3, %6, %9, 0 offen offset:2048 sc0 sc1\n\t"
                   "buffer_load_dwordx4 %4, %6, %9, 0 offen offset:3072 sc0 sc1\n\tbuffer_load_dwordx4 %5, %7, %9, 0 offen sc0 sc1\n\ts_waitcnt vmcnt(5)"
                   : "=&v"(mk), "=&v"(v[0]), "=&v"(v[1]), "=&v"(v[2]), "=&v"(v[3]), "=&v"(v[4]) : "v"(lo), "v"(hi), "s"(rm), "s"(re) : "memory");
    }
    if (early) lds_arrive(misc, lane);
    const unsigned cn = __builtin_popcount(mk[0]) + __builtin_popcount(mk[1]) + __builtin_popcount(mk[2]) + __builtin_popcount(mk[3]);
    const unsigned rn = cn + (cn >> 7) * 4u, incl = wave_scan(rn);
    const int pl = ((lane & 3) * 16 + (lane >> 2)) * 4;
#pragma unroll
    for (int w = 0; w < 4; ++w) mk[w] = __builtin_amdgcn_ds_bpermute(pl, mk[w]);
    unsigned adr[32], sel[32], pos = wv * QBLK + __builtin_amdgcn_ds_bpermute(pl, incl - rn);
#pragma unroll
    for (int k = 0; k < 32; ++k) {
      const unsigned nib = (mk[k >> 3] >> (4 * (k & 7))) & 15u;
      adr[k] = pos & ~3u;
      sel[k] = *(const LAS unsigned*)(lds + LUT + (((pos & 3u) << 4) | nib) * 4);
      pos += __builtin_popcount(nib);
    }
#pragma unroll
    for (int k = 0; k < 32; ++k) asm volatile("" : "+v"(adr[k]), "+v"(sel[k]));
    asm volatile("s_waitcnt vmcnt(0)" : "+v"(v[0]), "+v"(v[1]), "+v"(v[2]), "+v"(v[3]), "+v"(v[4]) :: "memory");
    const u32x4 tg = call_tag(call);
    auto take = [&](const u32x4& q, unsigned off) -> bool {
      *(LAS u32x4*)(lds + wv * QBLK + off) = q & 0x7f7f7f7fu;
      return (((q.x & 0x80808080u) == tg.x) & ((q.y & 0x80808080u) == tg.y) & ((q.z & 0x80808080u) == tg.z) & ((q.w & 0x80808080u) == tg.w)) | (off >= tot);
    };
    const rsrc_t ro = mkrsrc(P.out, MSH * NG);
    const unsigned obase = ((ts >> 3) * 256 + (wv >> 2) * 128 + (lane >> 2)) * NG + src * NBLK + (ts & 7) * 256 + (wv & 3) * 64 + (lane & 3) * 16;
    bool head = true;
#pragma unroll
    for (int i = 0; i < 5; ++i) head &= take(v[i], i * 1024 + lane * 16);
    head = __all(head);
    if (!early) {
      for (unsigned it = 1; !dead && (unsigned)(tam >> 32) != call; ++it) {
        if (wv != 0) __builtin_amdgcn_s_sleep(4);
        tam = uni64(__hip_atomic_load((const GAS u64*)(myws + TAM_OFF), __ATOMIC_RELAXED, __HIP_MEMORY_SCOPE_AGENT));
        if ((it & 63u) == 0u && give_up(tmo, call, t0)) dead = true;
      }
      while (!dead && !due()) __builtin_amdgcn_s_sleep(1);
    }
    bool passed = early;
    for (unsigned it = 1; !dead; ++it) {
      bool ok = true;
      const unsigned tv = ld_agent(tmo);
      if (!head) {
#pragma unroll
        for (int i = 0; i < 5; ++i) v[i] = LD128S(rq, i * 1024 + lane * 16);
        ok = take(v[0], lane * 16) & take(v[1], 1024 + lane * 16);
        if (!passed && __all(ok)) { passed = true; lds_arrive(misc, lane); }
#pragma unroll
        for (int i = 2; i < 5; ++i) ok &= take(v[i], i * 1024 + lane * 16);
      }
      if (tot > 5120u) {
#pragma unroll
        for (int i = 0; i < 4; ++i) v[i] = LD128S(rq, (5 + i) * 1024 + lane * 16);
#pragma unroll
        for (int i = 0; i < 4; ++i) ok &= take(v[i], (5 + i) * 1024 + lane * 16);
      }
      dead = __builtin_amdgcn_readfirstlane(tv) == call;
      if (__all(ok)) break;
      if ((it & 63u) == 0u && give_up(tmo, call, t0)) dead = true;
      asm volatile("" ::: "memory");
    }
    if (!passed) lds_arrive(misc, lane);
    if (dead) {
#pragma unroll
      for (int m = 0; m < 8; ++m) ST128(((u32x4){0x80808080u, 0x80808080u, 0x80808080u, 0x80808080u}), ro, obase + m * 16u * NG, SYS);
    } else {
      u32x2 pr[32];
#pragma unroll
      for (int k = 0; k < 32; ++k) { const LAS unsigned* p = (const LAS unsigned*)(lds + adr[k]); pr[k] = (u32x2){p[0], p[1]}; }
      __builtin_amdgcn_sched_barrier(0);
#pragma unroll
      for (int m = 0; m < 8; ++m) {
        u32x4 q;
#pragma unroll
        for (int j = 0; j < 4; ++j) q[j] = __builtin_amdgcn_perm(pr[m * 4 + j].y, pr[m * 4 + j].x, sel[m * 4 + j]);
        ST128(q, ro, obase + m * 16u * NG, SYS);
      }
    }
    return;
  }

  const unsigned sl = slot - NPUSH / 8, own = sl >> 1, ti = 2 * own + (xg & 1), tj = 2 * (xg >> 1) + (sl & 1);
  const int wr = wv >> 2, wc = wv & 3;
  const rsrc_t rs = mkrsrc(myws, DATA_BYTES);
#define BLDS(voff, so, dstoff, imm) __builtin_amdgcn_raw_ptr_buffer_load_lds(rs, (LAS void*)(lds + (dstoff)), 16, voff, so, imm, SYS)
#define STAGE1(so, lb) do { BLDS(gx, so, (lb), 0); BLDS(gx, so, (lb), 1024); BLDS(gw, so, (lb) + PIECE, 0); BLDS(gw, so, (lb) + PIECE, 1024); } while (0)
#define RD(X, W, h) do { _Pragma("unroll") for (int t = 0; t < 4; ++t) W[t] = *(const LAS bf16x8*)(lds + wa + (h) * 2 * PIECE + t * 1024); \
    _Pragma("unroll") for (int m = 0; m < 8; ++m) X[m] = *(const LAS bf16x8*)(lds + xa + (h) * 2 * PIECE + m * 1024); } while (0)
#define MMR(X, W, m0, m1) do { _Pragma("unroll") for (int m = (m0); m < (m1); ++m) _Pragma("unroll") for (int t = 0; t < 4; ++t) \
    acc[m][t] = __builtin_amdgcn_mfma_f32_16x16x32_bf16(W[t], X[m], acc[m][t], 0, 0, 0); } while (0)
#define SGB(m, n) __builtin_amdgcn_sched_group_barrier(m, n, 0)
  bool dead = false;
  int lane = lane_id();
  unsigned fo = FLAG_OFF + ((rank + 1 + (lane >> 2)) & 7) * FLAG_SRC + (((lane >> 1) & 1) * 24 + ((lane & 1) ? 16 + tj : ti)) * 128;
  unsigned sv = lane < 32 ? ((rank + 1 + (lane >> 2)) & 7) * SRC_BYTES + ((lane >> 1) & 1) * UNIT_BYTES : DATA_BYTES;
  const unsigned ls = lane < 32 ? lane >> 2 : 255;
  const unsigned gx = ti * PIECE + wv * 2048 + lane * 16, gw = (16 + tj) * PIECE + wv * 2048 + lane * 16;
  f32x4 acc[8][4];
  bf16x8 X0[8], W0[4], X1[8], W1[4];
  unsigned xa = wr * 8192 + (lane & 15) * 64 + (((lane >> 4) ^ (-((lane & 15) >> 2) & 3)) * 16), wa = PIECE + wc * 4096 + lane * 16;
  unsigned lb = wv * 2048;
  unsigned empties = 0, sn = 2 * BUF + 512;
  unsigned sna = sn + min(lane, 32) * 4;
  const u64 tw = __builtin_amdgcn_s_memrealtime();
  unsigned s0, s1, got, topick = 128;
  auto decide = [&](unsigned m) {
    m &= m >> 1;
    const unsigned r = m & 0x11111111u, h0 = __builtin_ctzll(r | 0x100000000ull);
    const unsigned r1 = (r & (r - 1u)) | (m & ((r & -r) << 2)), h1 = __builtin_ctzll(r1 | 0x100000000ull);
    s0 = __builtin_amdgcn_readlane(sv, h0); s1 = __builtin_amdgcn_readlane(sv, h1);
    const unsigned inc = (ls == h0 >> 2) + (ls == h1 >> 2);
    sv += inc * UNIT_BYTES; fo += inc * (24 * 128);
    got = 2u - (h0 >> 5) - (h1 >> 5); topick -= got;
  };
#define POLL() do { if (wv == 0 && lane < 32) __builtin_amdgcn_global_load_lds((const GAS unsigned*)(myws + fo), (LAS unsigned*)(lds + sn), 4, 0, SYS); } while (0)
#define SNAP() (*(const LAS unsigned*)(lds + sna))
#define LGKM0() __builtin_amdgcn_s_waitcnt(0xC07F)
#pragma unroll
  for (int m = 0; m < 8; ++m)
#pragma unroll
    for (int t = 0; t < 4; ++t) acc[m][t] = (f32x4){0.f, 0.f, 0.f, 0.f};
#pragma unroll
  for (int m = 0; m < 8; ++m) X0[m] = (bf16x8){0, 0, 0, 0, 0, 0, 0, 0};
#pragma unroll
  for (int t = 0; t < 4; ++t) W0[t] = (bf16x8){0, 0, 0, 0, 0, 0, 0, 0};
  unsigned cur = 0, nx = 0;
  for (;;) {
    if (cur == 0u) {
      bool quit = false;
      while (nx == 0u) {
        POLL();
        if (wv == 0 && (++empties & 63u) == 0u && give_up(tmo, call, tw)) {
          dead = true;
          if (lane == 0) *(LAS unsigned*)(lds + sn + 128) = ~0u;
        }
        WAIT_ALL();
        BAR();
        const u64 bal = __ballot(SNAP() >= call);
        quit = (bal >> 32) != 0ull;
        if (quit) break;
        decide((unsigned)bal);
        if (got > 0u) STAGE1(s0, lb ^ BUF);
        if (got > 1u) STAGE1(s1, (lb ^ BUF) + 2 * PIECE);
        sn ^= 256u; sna ^= 256u;
        nx = got;
      }
      if (quit) break;
      POLL();
      WAIT_ALL();
      BAR();
      xa ^= BUF; wa ^= BUF;
      decide((unsigned)__ballot(SNAP() >= call));
      if (got > 0u) STAGE1(s0, lb);
      if (got > 1u) STAGE1(s1, lb + 2 * PIECE);
      RD(X0, W0, 0);
      lb ^= BUF; sn ^= 256u; sna ^= 256u;
      cur = nx; nx = got;
    }
    RD(X1, W1, 1);
    MMR(X0, W0, 0, 6);
    xa ^= BUF; wa ^= BUF;
#pragma unroll
    for (int k = 0; k < 12; ++k) { SGB(0x008, 2); SGB(0x100, 1); }
    __builtin_amdgcn_sched_barrier(0);
    const unsigned want = topick != 0u ? call : ~0u;
    if (topick != 0u) POLL();
    MMR(X0, W0, 6, 8);
    __builtin_amdgcn_sched_barrier(0);
    if (cur + nx + topick == 1u) break;
    WAIT_ALL();
    BAR();
    __builtin_amdgcn_sched_barrier(0);
    if (cur == 2u) {
      LGKM0();
      __builtin_amdgcn_sched_barrier(0);
      const unsigned fv = SNAP();
      RD(X0, W0, 0);
      decide((unsigned)__ballot(fv >= want));
      STAGE1(s0, lb); STAGE1(s1, lb + 2 * PIECE);
      MMR(X1, W1, 0, 8);
      lb ^= BUF; sn ^= 256u; sna ^= 256u;
      cur = nx; nx = got;
      SGB(0x100, 1);
#pragma unroll
      for (int k = 0; k < 12; ++k) { SGB(0x008, 1); SGB(0x100, 1); }
      SGB(0x008, 2);
      SGB(0x002, 1); SGB(0x004, 4); SGB(0x008, 1); SGB(0x004, 6); SGB(0x008, 1); SGB(0x002, 2);
#pragma unroll
      for (int k = 0; k < 8; ++k) { SGB(0x020, 1); SGB(0x008, 2); }
      SGB(0x008, 8);
      __builtin_amdgcn_sched_barrier(0);
      asm volatile("" : "+v"(sv), "+v"(fo), "+v"(sna), "+s"(lb), "+s"(sn), "+s"(cur), "+s"(nx), "+s"(topick));
      if (cur + nx + topick == 0u) break;
    } else {
      asm volatile("" ::: "memory");
      decide((unsigned)__ballot(SNAP() >= want));
      if (got > 0u) STAGE1(s0, lb);
      if (got > 1u) STAGE1(s1, lb + 2 * PIECE);
      RD(X0, W0, 0);
      lb ^= BUF; sn ^= 256u; sna ^= 256u;
      cur = nx; nx = got;
    }
  }
  __builtin_amdgcn_s_setprio(2);
  asm volatile("" : "+v"(lane));
  float mx[4] = {0.f, 0.f, 0.f, 0.f};
#pragma unroll
  for (int m = 0; m < 8; ++m)
#pragma unroll
    for (int t = 0; t < 4; ++t) { mx[t] = __builtin_fmaxf(__builtin_fmaxf(mx[t], acc[m][t][0]), acc[m][t][1]); mx[t] = __builtin_fmaxf(__builtin_fmaxf(mx[t], acc[m][t][2]), acc[m][t][3]); }
  {
    const unsigned wm = wave_umax(__float_as_uint(__builtin_fmaxf(__builtin_fmaxf(mx[0], mx[1]), __builtin_fmaxf(mx[2], mx[3]))));
    WAIT_V(0);
    if (lane == 0) {
      unsigned old, mb;
      const unsigned ma = (unsigned)(size_t)misc, one = 1u;
      asm volatile("ds_max_u32 %2, %3\n\tds_add_rtn_u32 %0, %2, %4 offset:4\n\tds_read_b32 %1, %2\n\ts_waitcnt lgkmcnt(0)"
                   : "=&v"(old), "=&v"(mb) : "v"(ma), "v"(wm), "v"(one) : "memory");
      if (old == 7u)
        __hip_atomic_store((GAS u64*)(myws + LMAX_OFF + (ti * 8 + tj) * 8), ((u64)call << 32) | mb, __ATOMIC_RELAXED, __HIP_MEMORY_SCOPE_AGENT);
    }
    __builtin_amdgcn_s_setprio(0);
  }
  auto wait_scale = [&]() -> float {
    const char* gp = myws + (lane < 8 ? GMAX_OFF + lane * 128 : TMO_OFF);
    const u64 t0 = __builtin_amdgcn_s_memrealtime();
    unsigned mb = 0;
    for (unsigned it = 1;; ++it) {
      const u64 g = __hip_atomic_load((const GAS u64*)gp, __ATOMIC_RELAXED, __HIP_MEMORY_SCOPE_SYSTEM);
      mb = lane < NR ? (unsigned)g : 0u;
      dead = dead || __builtin_amdgcn_readlane((unsigned)g, 8) == call;
      if (dead || ((unsigned)__ballot((unsigned)(g >> 32) == call) & 0xffu) == 0xffu) break;
      if ((it & 63u) == 0u && give_up(tmo, call, t0)) { dead = true; break; }
    }
    mb = max(mb, DPP(mb, 0x111, 0xf)); mb = max(mb, DPP(mb, 0x112, 0xf)); mb = max(mb, DPP(mb, 0x114, 0xf));
    const float amax = __uint_as_float(__builtin_amdgcn_readlane(mb, 7));
    return amax > 0.f ? 127.0f / amax : 0.f;
  };
  constexpr float RND = 12582912.0f;
  if (own == rank) {
    const float inv = wait_scale();
    const rsrc_t ro = mkrsrc(P.out, MSH * NG);
    const unsigned obase = ((ti & 1) * 256 + wr * 128 + (lane & 15)) * NG + rank * NBLK + tj * 256 + wc * 64 + (lane >> 4) * 16;
#pragma unroll
    for (int m = 0; m < 8; ++m) {
      u32x4 q;
#pragma unroll
      for (int j = 0; j < 4; ++j) {
        unsigned by[4];
#pragma unroll
        for (int t = 0; t < 4; ++t) by[t] = __float_as_uint(__builtin_fmaf(__builtin_fmaxf(acc[m][t][j], 0.f), inv, RND));
        q[j] = dead ? 0x80808080u : __builtin_amdgcn_perm(by[1], by[0], 0x0c0c0400u) | __builtin_amdgcn_perm(by[3], by[2], 0x04000c0cu);
      }
      ST128(q, ro, obase + m * 16u * NG, SYS);
    }
  } else {
    const unsigned ts = (ti & 1) * 8 + tj, blk = own * 128 + ts * 8 + wv;
    u32x4 mk;
#pragma unroll
    for (int w = 0; w < 4; ++w) {
      unsigned a = 0;
#pragma unroll
      for (int k = 7; k >= 0; --k)
#pragma unroll
        for (int t = 3; t >= 0; --t) a = __builtin_amdgcn_alignbit(a, __float_as_uint(acc[2 * w + (k >> 2)][t][k & 3]), 31);
      mk[w] = ~a;
    }
    ST128(mk, mkrsrc(myws + MASK_OFF + blk * MASK_BLK, MASK_BLK), lane * 16, SYS);
    const unsigned cn = __builtin_popcount(mk[0]) + __builtin_popcount(mk[1]) + __builtin_popcount(mk[2]) + __builtin_popcount(mk[3]);
    const unsigned rn = cn + (cn >> 7) * 4u, incl = wave_scan(rn), tot = __builtin_amdgcn_readlane(incl, 63);
    WAIT_V(0);
    if (lane == 0) __hip_atomic_store((GAS u64*)(myws + MFLG_OFF + (own * 16 + ts) * 128 + wv * 8), ((u64)call << 32) | (tot + 2u), __ATOMIC_RELAXED, __HIP_MEMORY_SCOPE_SYSTEM);
    unsigned adr[32], pos = wv * QBLK + incl - rn;
    const unsigned fin = (pos + cn) & ~3u, sink = wv * QBLK + 8704 + lane * 4;
    u32x2 sel[32];
#pragma unroll
    for (int k = 0; k < 32; ++k) {
      const unsigned nib = (mk[k >> 3] >> (4 * (k & 7))) & 15u;
      adr[k] = pos >= fin ? sink : pos & ~3u;
      sel[k] = *(const LAS u32x2*)(lds + LUT + (((pos & 3u) << 4) | nib) * 8);
      pos += __builtin_popcount(nib);
    }
#pragma unroll
    for (int k = 0; k < 32; ++k) asm volatile("" : "+v"(adr[k]), "+v"(sel[k]));
    const rsrc_t rq = mkrsrc(myws + QDAT_OFF + blk * QBLK, (tot + 15u) & ~15u);
    const u32x4 tg = call_tag(call);
    if (wv < 4) __builtin_amdgcn_s_setprio(1);
    const float inv = wait_scale();
    if (!dead) {
      unsigned pend = 0;
      *(LAS unsigned*)(lds + fin) = 0u;
      asm volatile("" ::: "memory");
#pragma unroll
      for (int k = 0; k < 32; ++k) {
        if ((k & 3) == 0) {
#pragma unroll
          for (int t = 0; t < 4; ++t) acc[k >> 2][t] = __builtin_elementwise_fma(acc[k >> 2][t], (f32x4){inv, inv, inv, inv}, (f32x4){RND, RND, RND, RND});
        }
        unsigned by[4];
#pragma unroll
        for (int t = 0; t < 4; ++t) by[t] = __float_as_uint(acc[k >> 2][t][k & 3]);
        const unsigned d = __builtin_amdgcn_perm(by[1], by[0], 0x0c0c0400u) | __builtin_amdgcn_perm(by[3], by[2], 0x04000c0cu);
        *(LAS unsigned*)(lds + adr[k]) = __builtin_amdgcn_perm(d, pend, sel[k].x);
        pend = __builtin_amdgcn_perm(d, pend, sel[k].y);
      }
      asm volatile("" ::: "memory");
      __hip_atomic_fetch_or((LAS unsigned*)(lds + fin), pend, __ATOMIC_RELAXED, __HIP_MEMORY_SCOPE_WORKGROUP);
      WAIT_L0();
      u32x4 v[5];
#pragma unroll
      for (int i = 0; i < 5; ++i) v[i] = *(const LAS u32x4*)(lds + wv * QBLK + i * 1024 + lane * 16);
#pragma unroll
      for (int i = 0; i < 5; ++i) ST128(((v[i] & 0x7f7f7f7fu) | tg), rq, i * 1024 + lane * 16, SYS);
      if (tot > 5120u) {
#pragma unroll
        for (int i = 0; i < 4; ++i) v[i] = *(const LAS u32x4*)(lds + wv * QBLK + (5 + i) * 1024 + lane * 16);
#pragma unroll
        for (int i = 0; i < 4; ++i) ST128(((v[i] & 0x7f7f7f7fu) | tg), rq, (5 + i) * 1024 + lane * 16, SYS);
      }
    }
  }
}

}

extern "C" void kernel_launch(int rank, int nranks, ncclComm_t comm, hipStream_t stream,
                              void* const* d_in, const int* in_sizes, int n_in,
                              void* d_out, int out_size,
                              const int* in_dtypes, int out_dtype,
                              void* const* d_out_all,
                              void* const* d_ws_all, size_t ws_bytes) {
  if (nranks != NR || (unsigned)rank >= (unsigned)NR || n_in != 2 || in_sizes[0] != MG * KSH || in_sizes[1] != KSH * NG ||
      out_size != MSH * NG || ws_bytes < WS_NEED) {
    fprintf(stderr, "unexpected problem shape\n");
    return;
  }
  int dev = 0, cus = 0;
  HIP_CHECK(hipGetDevice(&dev));
  HIP_CHECK(hipDeviceGetAttribute(&cus, hipDeviceAttributeMultiprocessorCount, dev));
  if (cus < NWG) { fprintf(stderr, "needs %d CUs\n", NWG); return; }
  Params P;
  P.x = (const float*)d_in[0]; P.w = (const float*)d_in[1]; P.out = (char*)d_out; P.rank = rank; P.pad = 0;
  for (int k = 0; k < NR; ++k) P.ws[k] = (char*)d_ws_all[(rank + k) & 7];
  hipLaunchKernelGGL(fused_gemm_q8, dim3(NWG), dim3(512), 0, stream, P);
  HIP_CHECK(hipGetLastError());
}
```
